```python
import numpy as np

kernel_name = "gpu_gemm_rs_m4096_k4096_n16384_e4m3e4m3_dequant"


ABSMAX_RTOL = 0.03

COMPUTE_DTYPE = "float32"
EPILOGUE = "dequant"
IN_QUANT = None
OUT_QUANT = None
IN_FP8 = "e4m3e4m3"
IN_DTYPES = {"x": "e4m3", "w_mat": "e4m3",
             "scale_x": "float32", "scale_w": "float32"}
WORLD_SIZE = 8

def _snap_fp8(x, mant_bits, max_abs, min_normal):
    a = np.abs(x.astype(np.float64))
    _m, _e = np.frexp(a)
    q = np.where(a >= min_normal, np.ldexp(1.0, _e - 1 - mant_bits),
                 min_normal * 2.0 ** -mant_bits)
    snapped = np.minimum(np.round(a / q) * q, max_abs)
    return (np.sign(x) * snapped).astype(np.float32)

def setup_inputs():

    x = _snap_fp8(np.random.randn(4096, 4096).astype(np.float32), 3, 448.0, 2.0 ** -6)
    w_mat = _snap_fp8(np.random.randn(4096, 16384).astype(np.float32), 3, 448.0, 2.0 ** -6)
    scale_x = (0.02 * (1.0 + np.random.rand(1))).astype(np.float32)
    scale_w = (0.02 * (1.0 + np.random.rand(1))).astype(np.float32)
    return {"x": x, "w_mat": w_mat, "scale_x": scale_x, "scale_w": scale_w}

def reference(x, w_mat, scale_x, scale_w):
    acc = (x @ w_mat).astype(np.float32)
    y = (acc * (scale_x * scale_w)).astype(np.float32)
    return y
```

```cpp
#include <hip/hip_runtime.h>
#include <rccl/rccl.h>
#include <cstdio>
#include <cstdint>

#define HIP_CHECK(expr) do { hipError_t _e = (expr); if (_e != hipSuccess) { \
    fprintf(stderr, "HIP error %s at %s:%d\n", hipGetErrorString(_e), __FILE__, __LINE__); \
    return; } } while (0)

namespace {

typedef __attribute__((ext_vector_type(4))) unsigned u32x4;
typedef __attribute__((ext_vector_type(8))) int i32x8;
typedef __attribute__((ext_vector_type(4))) float f32x4;

constexpr int NR = 8;
constexpr int MM = 4096, NN = 16384;
constexpr int KS = 512;
constexpr int NS = 2048;
constexpr int G = 4;
constexpr int NPULL = (NR - 1) * G;
constexpr int GRID = 256;
constexpr int NGEMM = GRID - NPULL;
constexpr int NTHR = 256;

constexpr size_t MiB = 1ull << 20;
constexpr size_t OFF_CALLCNT = 0;
constexpr size_t OFF_FLAGS = 4096;
constexpr size_t OFF_LAND = 8192;
constexpr size_t OFF_TMO = 12288;
constexpr size_t OFF_XPUB = 1 * MiB;
constexpr size_t OFF_WPUB = 4 * MiB;
constexpr size_t OFF_XLAND = 16 * MiB;
constexpr size_t OFF_WLAND = 32 * MiB;
constexpr size_t OFF_OUT = 64 * MiB;
constexpr size_t WS_NEED = 96 * MiB;

constexpr int FLAG_W = 0, FLAG_X = 16, FLAG_O = 48;
constexpr unsigned long long DEADLINE_TICKS = 50000000ull;

struct Params {
  const unsigned char* x;
  const unsigned char* w;
  const float* sx;
  const float* sw;
  float* out;
  unsigned char* ws[NR];
  int rank;
};

#define WAIT_VM0() asm volatile("s_waitcnt vmcnt(0)" ::: "memory")

__device__ __forceinline__ unsigned ld_sys(const unsigned* p) { return __hip_atomic_load(p, __ATOMIC_RELAXED, __HIP_MEMORY_SCOPE_SYSTEM); }
__device__ __forceinline__ unsigned ld_agent(const unsigned* p) { return __hip_atomic_load(p, __ATOMIC_RELAXED, __HIP_MEMORY_SCOPE_AGENT); }

__device__ __forceinline__ bool wait_ge(const unsigned* p, unsigned target, unsigned* tmo, unsigned call) {
  unsigned long long t0 = __builtin_amdgcn_s_memrealtime();
  while (ld_sys(p) < target) {
    if (ld_agent(tmo) == call || __builtin_amdgcn_s_memrealtime() - t0 > DEADLINE_TICKS) {
      __hip_atomic_store(tmo, call, __ATOMIC_RELAXED, __HIP_MEMORY_SCOPE_AGENT);
      return false;
    }
    __builtin_amdgcn_s_sleep(2);
  }
  return true;
}

__device__ __forceinline__ void transpose4(unsigned r0, unsigned r1, unsigned r2, unsigned r3,
                                           unsigned& c0, unsigned& c1, unsigned& c2, unsigned& c3) {
  unsigned t0 = __builtin_amdgcn_perm(r1, r0, 0x05010400u);
  unsigned t1 = __builtin_amdgcn_perm(r1, r0, 0x07030602u);
  unsigned t2 = __builtin_amdgcn_perm(r3, r2, 0x05010400u);
  unsigned t3 = __builtin_amdgcn_perm(r3, r2, 0x07030602u);
  c0 = __builtin_amdgcn_perm(t2, t0, 0x05040100u);
  c1 = __builtin_amdgcn_perm(t2, t0, 0x07060302u);
  c2 = __builtin_amdgcn_perm(t3, t1, 0x05040100u);
  c3 = __builtin_amdgcn_perm(t3, t1, 0x07060302u);
}

__device__ __forceinline__ void copy_lin(const unsigned char* src, unsigned char* dst, int nbytes) {
  const u32x4* s = (const u32x4*)src; u32x4* d = (u32x4*)dst;
  const int n16 = nbytes >> 4;
  for (int i = threadIdx.x; i < n16; i += 4 * NTHR) {
    u32x4 v0 = s[i], v1 = s[i + NTHR], v2 = s[i + 2 * NTHR], v3 = s[i + 3 * NTHR];
    d[i] = v0; d[i + NTHR] = v1; d[i + 2 * NTHR] = v2; d[i + 3 * NTHR] = v3;
  }
}

__global__ void __launch_bounds__(NTHR, 1) fused_kernel(Params p) {
  __shared__ __attribute__((aligned(16))) unsigned char lds[131072 + 64];
  volatile unsigned* sctl = (volatile unsigned*)(lds + 131072);
#define s_call sctl[0]
#define s_ok sctl[1]
  const int tid = threadIdx.x, lane = tid & 63;
  const int wid = __builtin_amdgcn_readfirstlane(tid >> 6);
  const int b = blockIdx.x;
  const int r = p.rank;
  unsigned char* ws = p.ws[r];
  unsigned* tmo = (unsigned*)(ws + OFF_TMO);
  unsigned* land = (unsigned*)(ws + OFF_LAND);

  if (tid == 0) {
    unsigned* cc = (unsigned*)(ws + OFF_CALLCNT) + b;
    unsigned c = ld_agent(cc) + 1;
    __hip_atomic_store(cc, c, __ATOMIC_RELAXED, __HIP_MEMORY_SCOPE_AGENT);
    s_call = c; s_ok = 1;
  }
  __syncthreads();
  const unsigned call = s_call;

  if (b < NPULL) {
    const int pi = b % (NR - 1), g = b / (NR - 1);
    const unsigned src = (unsigned)(r + 1 + pi) % NR;
    if (src >= (unsigned)NR) return;
    const unsigned char* rws = p.ws[src];
    const unsigned* fl = (const unsigned*)(ws + OFF_FLAGS) + src * 128;
    if (tid == 0) {
      bool ok = true;
      for (int i = 0; i < 4; ++i) ok = ok && wait_ge(fl + FLAG_W + g * 4 + i, call, tmo, call);
      __builtin_amdgcn_fence(__ATOMIC_ACQUIRE, "");
      if (!ok) s_ok = 0;
    }
    __syncthreads();
    copy_lin(rws + OFF_WPUB + (size_t)r * MiB + (size_t)g * 262144, ws + OFF_WLAND + (size_t)src * MiB + (size_t)g * 262144, 262144);
    WAIT_VM0();
    __syncthreads();
    if (tid == 0) {
      __builtin_amdgcn_fence(__ATOMIC_RELEASE, "agent");
      WAIT_VM0();
      __hip_atomic_fetch_add(land + 0, 1u, __ATOMIC_RELAXED, __HIP_MEMORY_SCOPE_AGENT);
    }
    for (int j = 0; j < 4; ++j) {
      if (tid == 0) {
        bool ok = true;
        for (int i = 0; i < 2; ++i) ok = ok && wait_ge(fl + FLAG_X + (2 * g + i) * 4 + j, call, tmo, call);
        __builtin_amdgcn_fence(__ATOMIC_ACQUIRE, "");
        if (!ok) s_ok = 0;
      }
      __syncthreads();
      for (int i = 0; i < 2; ++i) {
        const size_t o = (size_t)((2 * g + i) * 4 + j) * 65536;
        copy_lin(rws + OFF_XPUB + o, ws + OFF_XLAND + (size_t)src * 2 * MiB + o, 65536);
      }
      WAIT_VM0();
      __syncthreads();
      if (tid == 0) {
        __builtin_amdgcn_fence(__ATOMIC_RELEASE, "agent");
        WAIT_VM0();
        __hip_atomic_fetch_add(land + 32 * (1 + j), 1u, __ATOMIC_RELAXED, __HIP_MEMORY_SCOPE_AGENT);
      }
    }
    for (int j = 0; j < 4; ++j) {
      for (int i = 0; i < 4; ++i) {
        const int nt = g * 4 + i;
        if (tid == 0) {
          bool ok = wait_ge(fl + FLAG_O + j * 16 + nt, call, tmo, call);
          __builtin_amdgcn_fence(__ATOMIC_ACQUIRE, "");
          if (!ok || ld_agent(tmo) == call) s_ok = 0;
        }
        __syncthreads();
        const bool ok = s_ok != 0;
        const u32x4* s = (const u32x4*)(rws + OFF_OUT + ((size_t)(r * 4 + j) * 16 + nt) * 65536);
        float* dbase = p.out + (size_t)(j * 128) * NN + (size_t)src * NS + nt * 128;
        for (int it = tid; it < 4096; it += 4 * NTHR) {
          u32x4 v[4];
          if (ok) {
#pragma unroll
            for (int u = 0; u < 4; ++u) v[u] = s[it + u * NTHR];
          } else {
#pragma unroll
            for (int u = 0; u < 4; ++u) v[u] = (u32x4){0x7fc00000u, 0x7fc00000u, 0x7fc00000u, 0x7fc00000u};
          }
#pragma unroll
          for (int u = 0; u < 4; ++u) {
            const int q = it + u * NTHR;
            *(u32x4*)(dbase + (size_t)(q >> 5) * NN + (q & 31) * 4) = v[u];
          }
        }
        __syncthreads();
      }
    }
    return;
  }

  const int gb = b - NPULL;
  if (gb < 160) {
    if (gb < 128) {
      const int kb = gb & 3, nq = gb >> 2, d = nq >> 2, ntq = nq & 3;
      const int ng = tid & 31, kg = tid >> 5;
      const unsigned char* sp = p.w + (size_t)(kb * 128 + kg * 16) * NN + nq * 512 + ng * 16;
      u32x4 in[16];
#pragma unroll
      for (int i = 0; i < 16; ++i) in[i] = *(const u32x4*)(sp + (size_t)i * NN);
      u32x4 o[16];
#pragma unroll
      for (int v = 0; v < 4; ++v)
#pragma unroll
        for (int w = 0; w < 4; ++w) {
          unsigned c0, c1, c2, c3;
          transpose4(in[4 * v][w], in[4 * v + 1][w], in[4 * v + 2][w], in[4 * v + 3][w], c0, c1, c2, c3);
          o[4 * w][v] = c0; o[4 * w + 1][v] = c1; o[4 * w + 2][v] = c2; o[4 * w + 3][v] = c3;
        }
      const int ntl = ng >> 3, nb = ng & 7, h = kg >> 2, q = kg & 3;
      const int base16 = ntl * 1024 + nb * 128 + h * 64 + q * 16;
#pragma unroll
      for (int c = 0; c < 16; ++c) *(u32x4*)(lds + (size_t)((base16 + c) ^ nb) * 16) = o[c];
      __syncthreads();
      unsigned char* dp = ws + OFF_WPUB + (size_t)d * MiB;
#pragma unroll
      for (int i = 0; i < 16; ++i) {
        const int L = i * NTHR + tid;
        u32x4 v = *(const u32x4*)(lds + (size_t)(L ^ ((L >> 7) & 7)) * 16);
        const int nt = ntq * 4 + (L >> 10);
        *(u32x4*)(dp + (size_t)(nt * 4 + kb) * 16384 + (size_t)(L & 1023) * 16) = v;
      }
      WAIT_VM0();
      __syncthreads();
      if (tid == 0) {
        if (d == r) {
          __builtin_amdgcn_fence(__ATOMIC_RELEASE, "agent");
          WAIT_VM0();
          __hip_atomic_fetch_add(land + 0, 1u, __ATOMIC_RELAXED, __HIP_MEMORY_SCOPE_AGENT);
        } else {
          __builtin_amdgcn_fence(__ATOMIC_RELEASE, "");
          WAIT_VM0();
          unsigned* f = (unsigned*)(p.ws[d] + OFF_FLAGS) + r * 128 + FLAG_W + ntq * 4 + kb;
          __hip_atomic_store(f, call, __ATOMIC_RELAXED, __HIP_MEMORY_SCOPE_SYSTEM);
        }
      }
    } else {
      const int mt = gb - 128;
      const u32x4* s = (const u32x4*)(p.x + (size_t)mt * 65536);
      u32x4* d = (u32x4*)(ws + OFF_XPUB + (size_t)mt * 65536);
      u32x4 v[16];
#pragma unroll
      for (int i = 0; i < 16; ++i) v[i] = s[i * NTHR + tid];
#pragma unroll
      for (int i = 0; i < 16; ++i) d[i * NTHR + tid] = v[i];
      WAIT_VM0();
      __syncthreads();
      if (tid == 0) {
        __builtin_amdgcn_fence(__ATOMIC_RELEASE, "");
        WAIT_VM0();
        for (int q = 1; q < NR; ++q) {
          const unsigned pr = (unsigned)(r + q) % NR;
          unsigned* f = (unsigned*)(p.ws[pr] + OFF_FLAGS) + r * 128 + FLAG_X + mt;
          __hip_atomic_store(f, call, __ATOMIC_RELAXED, __HIP_MEMORY_SCOPE_SYSTEM);
        }
      }
    }
    __syncthreads();
  }

  const float scale = p.sx[0] * p.sw[0];
  const int xcd = b & 7;
  const int first = NPULL + ((xcd - NPULL) & 7);
  const int cx = (GRID - first + 7) >> 3;
  const int li = (b - first) >> 3;
  const int wn = wid >> 1, wm = wid & 1;
  const int fq = lane >> 4, fc = lane & 15;
  bool wok = false;
  int jseen = -1;
  for (int t = li; t < 64; t += cx) {
    const int j = t >> 4, dd = (t >> 1) & 7, nt = xcd * 2 + (t & 1);
    const int d = (r + 1 + dd) & 7;
    const int mt = d * 4 + j;
    if (!wok || j > jseen) {
      if (tid == 0) {
        bool ok = true;
        if (!wok) ok = wait_ge(land + 0, call * (unsigned)(NPULL + 16), tmo, call);
        ok = ok && wait_ge(land + 32 * (1 + j), call * (unsigned)NPULL, tmo, call);
        __builtin_amdgcn_fence(__ATOMIC_ACQUIRE, "agent");
        WAIT_VM0();
        if (!ok) s_ok = 0;
      }
      __syncthreads();
      wok = true; jseen = j;
    }
    f32x4 acc[4][4];
#pragma unroll
    for (int i = 0; i < 4; ++i)
#pragma unroll
      for (int k = 0; k < 4; ++k) acc[i][k] = (f32x4){0.f, 0.f, 0.f, 0.f};

    auto issue = [&](int s) {
      const int src = (r + (s >> 2)) & 7, kb = s & 3;
      const unsigned char* wb = (src == r) ? ws + OFF_WPUB + (size_t)r * MiB : ws + OFF_WLAND + (size_t)src * MiB;
      const unsigned char* xb = (src == r) ? p.x : ws + OFF_XLAND + (size_t)src * 2 * MiB;
      unsigned char* lb = lds + (s & 3) * 32768;
      const unsigned char* wsrc = wb + (size_t)(nt * 4 + kb) * 16384 + wid * 4096 + lane * 16;
      const unsigned char* xsrc = xb + (size_t)(mt * 128 + wid * 32 + fc) * KS + kb * 128 + fq * 16;
#pragma unroll
      for (int i = 0; i < 4; ++i)
        __builtin_amdgcn_global_load_lds((const unsigned*)(wsrc + i * 1024), (unsigned*)(lb + wid * 4096 + i * 1024), 16, 0, 0);
#pragma unroll
      for (int i = 0; i < 4; ++i)
        __builtin_amdgcn_global_load_lds((const unsigned*)(xsrc + (i >> 1) * 16 * KS + (i & 1) * 64),
                                         (unsigned*)(lb + 16384 + wid * 4096 + i * 1024), 16, 0, 0);
    };
    auto compute = [&](int s) {
      const unsigned char* lb = lds + (s & 3) * 32768;
      i32x8 wf[4], xf[4];
#pragma unroll
      for (int i = 0; i < 4; ++i) {
        const u32x4 a0 = *(const u32x4*)(lb + (wn * 4 + i) * 2048 + lane * 16);
        const u32x4 a1 = *(const u32x4*)(lb + (wn * 4 + i) * 2048 + 1024 + lane * 16);
        wf[i] = (i32x8){(int)a0[0], (int)a0[1], (int)a0[2], (int)a0[3], (int)a1[0], (int)a1[1], (int)a1[2], (int)a1[3]};
        const u32x4 b0 = *(const u32x4*)(lb + 16384 + (wm * 4 + i) * 2048 + lane * 16);
        const u32x4 b1 = *(const u32x4*)(lb + 16384 + (wm * 4 + i) * 2048 + 1024 + lane * 16);
        xf[i] = (i32x8){(int)b0[0], (int)b0[1], (int)b0[2], (int)b0[3], (int)b1[0], (int)b1[1], (int)b1[2], (int)b1[3]};
      }
#pragma unroll
      for (int i = 0; i < 4; ++i)
#pragma unroll
        for (int k = 0; k < 4; ++k)
          acc[i][k] = __builtin_amdgcn_mfma_scale_f32_16x16x128_f8f6f4(wf[i], xf[k], acc[i][k], 0, 0, 0, 0x7F, 0, 0x7F);
    };
#define RAW_BAR() do { asm volatile("s_waitcnt lgkmcnt(0)" ::: "memory"); __builtin_amdgcn_s_barrier(); } while (0)
    issue(0); issue(1); issue(2);
    for (int s = 0; s < 30; ++s) {
      asm volatile("s_waitcnt vmcnt(16)" ::: "memory");
      RAW_BAR();
      if (s < 29) issue(s + 3);
      else asm volatile("" ::: "memory");
      compute(s);
    }
    asm volatile("s_waitcnt vmcnt(8)" ::: "memory");
    RAW_BAR();
    compute(30);
    asm volatile("s_waitcnt vmcnt(0)" ::: "memory");
    RAW_BAR();
    compute(31);
    RAW_BAR();

    const bool ok = s_ok != 0;
    float* ob; size_t ld;
    if (d == r) { ob = p.out + (size_t)(j * 128) * NN + (size_t)r * NS + nt * 128; ld = NN; }
    else { ob = (float*)(ws + OFF_OUT + ((size_t)(d * 4 + j) * 16 + nt) * 65536); ld = 128; }
#pragma unroll
    for (int i = 0; i < 4; ++i)
#pragma unroll
      for (int k = 0; k < 4; ++k) {
        f32x4 v = acc[i][k] * scale;
        if (!ok) v = (f32x4){__builtin_nanf(""), __builtin_nanf(""), __builtin_nanf(""), __builtin_nanf("")};
        const int m = (wm * 4 + k) * 16 + fc, n = (wn * 4 + i) * 16 + fq * 4;
        *(f32x4*)(ob + (size_t)m * ld + n) = v;
      }
    if (d != r) {
      WAIT_VM0();
      __syncthreads();
      if (tid == 0) {
        __builtin_amdgcn_fence(__ATOMIC_RELEASE, "");
        WAIT_VM0();
        unsigned* f = (unsigned*)(p.ws[d] + OFF_FLAGS) + r * 128 + FLAG_O + j * 16 + nt;
        __hip_atomic_store(f, call, __ATOMIC_RELAXED, __HIP_MEMORY_SCOPE_SYSTEM);
      }
    }
  }
}

}

extern "C" void kernel_launch(int rank, int nranks, ncclComm_t comm,
                              hipStream_t stream,
                              void* const* d_in, const int* in_sizes, int n_in,
                              void* d_out, int out_size,
                              const int* in_dtypes, int out_dtype,
                              void* const* d_out_all,
                              void* const* d_ws_all, size_t ws_bytes) {
  if (nranks != NR || ws_bytes < WS_NEED || rank < 0 || rank >= NR) {
    fprintf(stderr, "unexpected configuration\n");
    return;
  }
  Params p;
  p.x = (const unsigned char*)d_in[0];
  p.w = (const unsigned char*)d_in[1];
  p.sx = (const float*)d_in[2];
  p.sw = (const float*)d_in[3];
  p.out = (float*)d_out;
  for (int i = 0; i < NR; ++i) p.ws[i] = (unsigned char*)d_ws_all[i];
  p.rank = rank;
  hipLaunchKernelGGL(fused_kernel, dim3(GRID), dim3(NTHR), 0, stream, p);
  HIP_CHECK(hipGetLastError());
}
```

```cpp
#include <hip/hip_runtime.h>
#include <rccl/rccl.h>
#include <cstdio>
#include <cstdint>

#define HIP_CHECK(expr) do { hipError_t _e = (expr); if (_e != hipSuccess) { \
    fprintf(stderr, "HIP error %s at %s:%d\n", hipGetErrorString(_e), __FILE__, __LINE__); \
    return; } } while (0)

namespace {

typedef __attribute__((ext_vector_type(4))) unsigned u32x4;
typedef __attribute__((ext_vector_type(8))) int i32x8;
typedef __attribute__((ext_vector_type(4))) float f32x4;

constexpr int NR = 8;
constexpr int NN = 16384;
constexpr int KS = 512;
constexpr int NS = 2048;
constexpr int G = 4;
constexpr int NPULL = (NR - 1) * G;
constexpr int GRID = 256;
constexpr int NTHR = 256;

constexpr size_t MiB = 1ull << 20;
constexpr size_t OFF_CALLCNT = 0;
constexpr size_t OFF_FLAGS = 4096;
constexpr size_t OFF_LAND = 16384;
constexpr size_t OFF_QUEUE = 10240;
constexpr size_t OFF_TICKET = 11264;
constexpr size_t OFF_TMO = 12288;
constexpr size_t OFF_TRACE = 65536;
constexpr size_t OFF_XPUB = 1 * MiB;
constexpr size_t OFF_WPUB = 4 * MiB;
constexpr size_t OFF_XLAND = 16 * MiB;
constexpr size_t OFF_WLAND = 32 * MiB;
constexpr size_t OFF_OUT = 64 * MiB;
constexpr size_t WS_NEED = 96 * MiB;

constexpr int FLAG_W = 0, FLAG_X = 16, FLAG_O = 48;
constexpr unsigned NPIECE = 448;
constexpr unsigned long long DEADLINE_TICKS = 50000000ull;

struct Params {
  const unsigned char* x;
  const unsigned char* w;
  const float* sx;
  const float* sw;
  float* out;
  unsigned char* ws[NR];
  int rank;
};

#define WAIT_VM0() asm volatile("s_waitcnt vmcnt(0)" ::: "memory")
#define TR() do {} while (0)
#define TR2(t) do {} while (0)

constexpr int AUX_SYS = 17;
constexpr int AUX_AGENT = 16;

__device__ __forceinline__ unsigned ld_sys(const unsigned* p) { return __hip_atomic_load(p, __ATOMIC_RELAXED, __HIP_MEMORY_SCOPE_SYSTEM); }
__device__ __forceinline__ unsigned ld_agent(const unsigned* p) { return __hip_atomic_load(p, __ATOMIC_RELAXED, __HIP_MEMORY_SCOPE_AGENT); }

__device__ __forceinline__ __amdgpu_buffer_rsrc_t mkrsrc(const void* base) {
  unsigned long long a = (unsigned long long)base;
  unsigned lo = __builtin_amdgcn_readfirstlane((unsigned)a), hi = __builtin_amdgcn_readfirstlane((unsigned)(a >> 32));
  void* pb = (void*)(((unsigned long long)hi << 32) | lo);
  return __builtin_amdgcn_make_buffer_rsrc(pb, 0, 0x7fffffff, 0x00020000);
}

__device__ __forceinline__ bool wait_ge(const unsigned* p, unsigned target, unsigned* tmo, unsigned call) {
  unsigned long long t0 = __builtin_amdgcn_s_memrealtime();
  while (ld_sys(p) < target) {
    if (ld_agent(tmo) == call || __builtin_amdgcn_s_memrealtime() - t0 > DEADLINE_TICKS) {
      __hip_atomic_store(tmo, call, __ATOMIC_RELAXED, __HIP_MEMORY_SCOPE_AGENT);
      return false;
    }
    __builtin_amdgcn_s_sleep(2);
  }
  return true;
}

__device__ __forceinline__ void transpose4(unsigned r0, unsigned r1, unsigned r2, unsigned r3,
                                           unsigned& c0, unsigned& c1, unsigned& c2, unsigned& c3) {
  unsigned t0 = __builtin_amdgcn_perm(r1, r0, 0x05010400u);
  unsigned t1 = __builtin_amdgcn_perm(r1, r0, 0x07030602u);
  unsigned t2 = __builtin_amdgcn_perm(r3, r2, 0x05010400u);
  unsigned t3 = __builtin_amdgcn_perm(r3, r2, 0x07030602u);
  c0 = __builtin_amdgcn_perm(t2, t0, 0x05040100u);
  c1 = __builtin_amdgcn_perm(t2, t0, 0x07060302u);
  c2 = __builtin_amdgcn_perm(t3, t1, 0x05040100u);
  c3 = __builtin_amdgcn_perm(t3, t1, 0x07060302u);
}

__device__ __forceinline__ i32x8 ldfrag(const unsigned char* p) {
  const u32x4 a0 = *(const u32x4*)(p);
  const u32x4 a1 = *(const u32x4*)(p + 1024);
  return (i32x8){(int)a0[0], (int)a0[1], (int)a0[2], (int)a0[3], (int)a1[0], (int)a1[1], (int)a1[2], (int)a1[3]};
}

__global__ void __launch_bounds__(NTHR, 1) fused_kernel(Params p) {
  __shared__ __attribute__((aligned(16))) unsigned char lds[131072 + 64];
  volatile unsigned* sctl = (volatile unsigned*)(lds + 131072);
#define s_call sctl[0]
#define s_ok sctl[1]
#define s_tile sctl[2]
  const int tid = threadIdx.x, lane = tid & 63;
  const int wid = __builtin_amdgcn_readfirstlane(tid >> 6);
  const int b = blockIdx.x;
  const int r = p.rank;
  unsigned char* ws = p.ws[r];
  unsigned* tmo = (unsigned*)(ws + OFF_TMO);
  unsigned* land = (unsigned*)(ws + OFF_LAND);
  int trn = 0; (void)trn;
  TR();

  if (tid == 0) {
    unsigned* cc = (unsigned*)(ws + OFF_CALLCNT) + b;
    unsigned c = ld_agent(cc) + 1;
    __hip_atomic_store(cc, c, __ATOMIC_RELAXED, __HIP_MEMORY_SCOPE_AGENT);
    s_call = c; s_ok = 1;
  }

  if (b < NPULL) {
    __syncthreads();
    const unsigned call = __builtin_amdgcn_readfirstlane(s_call);
    const int pi = b % (NR - 1), g = b / (NR - 1);
    const unsigned src = (unsigned)(r + 1 + pi) % NR;
    if (src >= (unsigned)NR) return;
    const unsigned char* rws = p.ws[src];
    const unsigned* fl = (const unsigned*)(ws + OFF_FLAGS) + src * 128;
    const __amdgpu_buffer_rsrc_t rs_rem = mkrsrc(rws);
    const __amdgpu_buffer_rsrc_t rs_loc = mkrsrc(ws);
    const __amdgpu_buffer_rsrc_t rs_out = mkrsrc(p.out);
    bool ok = true;
    unsigned* pend = nullptr;
    unsigned* tk = (unsigned*)(ws + OFF_TICKET) + src * 32;
    const unsigned tbase = (call - 1) * (NPIECE + 4u * G);
    int zopq = 0; asm volatile("" : "+v"(zopq));
    tk += zopq;
    const unsigned* f1p = lane == 63 ? tmo : fl + 64 + lane;
    unsigned traw = 0, f0 = 0, f1 = 0;
    if (lane == 0) traw = __hip_atomic_fetch_add(tk, 1u, __ATOMIC_RELAXED, __HIP_MEMORY_SCOPE_AGENT);
    unsigned tn = __builtin_amdgcn_readfirstlane(traw) - tbase;
    unsigned long long rdy0 = 0, rdy1 = 0;
#define NEXT_STATE() do { tn = __builtin_amdgcn_readfirstlane(traw) - tbase; rdy0 = __ballot(f0 >= call); rdy1 = __ballot(lane != 63 && f1 >= call); \
      if (__builtin_amdgcn_readlane(f1, 63) == call) ok = false; } while (0)
    for (;;) {
      const unsigned t = tn;
      if (t >= NPIECE) break;
      if (lane == 0) traw = __hip_atomic_fetch_add(tk, 1u, __ATOMIC_RELAXED, __HIP_MEMORY_SCOPE_AGENT);
      unsigned fid, so, dof; unsigned* lc = nullptr;
      if (t < 32u || (t >= 96u && t < 192u)) {
        const unsigned v = t < 32u ? t : t - 96u + 32u;
        const unsigned j = v >> 5, mt = ((v & 31) >> 2) * 4 + j;
        const unsigned o = mt * 65536u + (v & 3) * 16384u;
        fid = FLAG_X + mt; so = (unsigned)OFF_XPUB + o; dof = (unsigned)(OFF_XLAND + (size_t)src * 2 * MiB) + o; lc = land + src * 256 + mt * 4 + (v & 3);
      } else if (t < 96u) {
        const unsigned u = t - 32u;
        fid = FLAG_W + (u >> 4) * 4 + (u & 3);
        so = (unsigned)(OFF_WPUB + (size_t)r * MiB) + u * 16384u; dof = (unsigned)(OFF_WLAND + (size_t)src * MiB) + u * 16384u; lc = land + src * 256 + 128 + u;
      } else {
        const unsigned v = t - 192u, j = v >> 6, nt = (v >> 2) & 15, qt = v & 3;
        fid = FLAG_O + j * 16 + nt;
        so = (unsigned)(OFF_OUT + ((size_t)(r * 4 + j) * 16 + nt) * 65536) + qt * 16384u;
        dof = (unsigned)(((size_t)(j * 128 + qt * 32) * NN + (size_t)src * NS + nt * 128) * 4);
      }
      if (!(((fid < 64u ? rdy0 >> fid : rdy1 >> (fid - 64u)) & 1ull))) {
        if (pend) { WAIT_VM0(); if (lane == 0) __hip_atomic_store(pend, call, __ATOMIC_RELAXED, __HIP_MEMORY_SCOPE_AGENT); pend = nullptr; }
        ok = wait_ge(fl + fid, call, tmo, call) && ok;
      }
      f0 = ld_sys(fl + lane); f1 = ld_sys(f1p);
      if (t == 0u || t == 32u || t == 96u || t == 128u || t == 160u || t == 192u || t == 256u || t == 320u || t == 384u) TR2(t);
      if (lc) {
#pragma unroll
        for (int bt = 0; bt < 4; ++bt) {
          const unsigned o = (unsigned)bt * 4096u + lane * 16u;
          u32x4 v[4];
#pragma unroll
          for (int u = 0; u < 4; ++u) v[u] = __builtin_amdgcn_raw_buffer_load_b128(rs_rem, so + o + u * 1024u, 0, AUX_SYS);
          WAIT_VM0();
          if (bt == 3) NEXT_STATE();
          if (pend) { if (lane == 0) __hip_atomic_store(pend, call, __ATOMIC_RELAXED, __HIP_MEMORY_SCOPE_AGENT); pend = nullptr; }
#pragma unroll
          for (int u = 0; u < 4; ++u) __builtin_amdgcn_raw_buffer_store_b128(v[u], rs_loc, dof + o + u * 1024u, 0, AUX_AGENT);
        }
        pend = lc;
      } else {
#pragma unroll
        for (int bt = 0; bt < 4; ++bt) {
          u32x4 v[4];
          if (ok) {
#pragma unroll
            for (int u = 0; u < 4; ++u) v[u] = __builtin_amdgcn_raw_buffer_load_b128(rs_rem, so + bt * 4096u + u * 1024u + lane * 16u, 0, AUX_SYS);
          } else {
#pragma unroll
            for (int u = 0; u < 4; ++u) v[u] = (u32x4){0x7fc00000u, 0x7fc00000u, 0x7fc00000u, 0x7fc00000u};
          }
          WAIT_VM0();
          const bool okn = ok; (void)okn;
          if (bt == 3) NEXT_STATE();
          if (pend) { if (lane == 0) __hip_atomic_store(pend, call, __ATOMIC_RELAXED, __HIP_MEMORY_SCOPE_AGENT); pend = nullptr; }
#pragma unroll
          for (int u = 0; u < 4; ++u) {
            const unsigned row = bt * 8 + u * 2 + (lane >> 5);
            __builtin_amdgcn_raw_buffer_store_b128(v[u], rs_out, dof + row * (unsigned)(NN * 4) + (lane & 31) * 16u, 0, 0);
          }
        }
      }
    }
    if (pend) { WAIT_VM0(); if (lane == 0) __hip_atomic_store(pend, call, __ATOMIC_RELAXED, __HIP_MEMORY_SCOPE_AGENT); }
    TR();
    return;
  }

  const int gb = b - NPULL;
  if (gb < 160) {
    const __amdgpu_buffer_rsrc_t rs_loc = mkrsrc(ws);
    if (gb >= 32) {
      const int u = gb - 32;
      const int kb = u & 3, nq = u >> 2, d = nq >> 2, ntq = nq & 3;
      const int ng = tid & 31, kg = tid >> 5;
      const unsigned char* sp = p.w + (size_t)(kb * 128 + kg * 16) * NN + nq * 512 + ng * 16;
      u32x4 in[16];
#pragma unroll
      for (int i = 0; i < 16; ++i) in[i] = *(const u32x4*)(sp + (size_t)i * NN);
      u32x4 o[16];
#pragma unroll
      for (int v = 0; v < 4; ++v)
#pragma unroll
        for (int w = 0; w < 4; ++w) {
          unsigned c0, c1, c2, c3;
          transpose4(in[4 * v][w], in[4 * v + 1][w], in[4 * v + 2][w], in[4 * v + 3][w], c0, c1, c2, c3);
          o[4 * w][v] = c0; o[4 * w + 1][v] = c1; o[4 * w + 2][v] = c2; o[4 * w + 3][v] = c3;
        }
      const int ntl = ng >> 3, nb = ng & 7, h = kg >> 2, q = kg & 3;
      const int base16 = ntl * 1024 + nb * 128 + h * 64 + q * 16;
#pragma unroll
      for (int c = 0; c < 16; ++c) *(u32x4*)(lds + (size_t)((base16 + c) ^ nb) * 16) = o[c];
      __syncthreads();
      const unsigned dof = (unsigned)(OFF_WPUB + (size_t)d * MiB);
#pragma unroll
      for (int i = 0; i < 16; ++i) {
        const int L = i * NTHR + tid;
        u32x4 v = *(const u32x4*)(lds + (size_t)(L ^ ((L >> 7) & 7)) * 16);
        const int nt = ntq * 4 + (L >> 10);
        __builtin_amdgcn_raw_buffer_store_b128(v, rs_loc, dof + (unsigned)(nt * 4 + kb) * 16384u + (unsigned)(L & 1023) * 16u, 0, AUX_SYS);
      }
      WAIT_VM0();
      __syncthreads();
      if (tid < 4) {
        const unsigned call = s_call;
        if (d == r) {
          __hip_atomic_store(land + r * 256 + 128 + (ntq * 4 + tid) * 4 + kb, call, __ATOMIC_RELAXED, __HIP_MEMORY_SCOPE_AGENT);
        } else if (tid == 0) {
          unsigned* f = (unsigned*)(p.ws[d] + OFF_FLAGS) + r * 128 + FLAG_W + ntq * 4 + kb;
          __hip_atomic_store(f, call, __ATOMIC_RELAXED, __HIP_MEMORY_SCOPE_SYSTEM);
        }
      }
    } else {
      const int mt = gb;
      u32x4 v[16];
#pragma unroll
      for (int i = 0; i < 16; ++i) {
        const int L = i * NTHR + tid;
        const int kb = L >> 10, mb = (L >> 7) & 7, h = (L >> 6) & 1, q = (L >> 4) & 3, c = L & 15;
        v[i] = *(const u32x4*)(p.x + (size_t)(mt * 128 + mb * 16 + c) * KS + kb * 128 + h * 64 + q * 16);
      }
#pragma unroll
      for (int i = 0; i < 16; ++i)
        __builtin_amdgcn_raw_buffer_store_b128(v[i], rs_loc, (unsigned)OFF_XPUB + (unsigned)mt * 65536u + (unsigned)(i * NTHR + tid) * 16u, 0, AUX_SYS);
      WAIT_VM0();
      __syncthreads();
      if (tid < NR + 4) {
        const unsigned call = s_call;
        if (tid >= NR) {
          __hip_atomic_store(land + r * 256 + mt * 4 + (tid - NR), call, __ATOMIC_RELAXED, __HIP_MEMORY_SCOPE_AGENT);
        } else if (tid != r) {
          unsigned* f = (unsigned*)(p.ws[tid] + OFF_FLAGS) + r * 128 + FLAG_X + mt;
          __hip_atomic_store(f, call, __ATOMIC_RELAXED, __HIP_MEMORY_SCOPE_SYSTEM);
        }
      }
    }
  }
  __syncthreads();
  const unsigned call = __builtin_amdgcn_readfirstlane(s_call);
  TR();

  const float scale = p.sx[0] * p.sw[0];
  const int xcd = b & 7;
  const int first = NPULL + ((xcd - NPULL) & 7);
  const unsigned cx = (unsigned)(GRID - first + 7) >> 3;
  unsigned* qh = (unsigned*)(ws + OFF_QUEUE) + xcd * 32;
  const int wn = wid >> 1, wm = wid & 1;
  const int fq = lane >> 4, fc = lane & 15;
  for (;;) {
    if (wid == 0) {
      unsigned traw = 0;
      if (lane == 0) traw = __hip_atomic_fetch_add(qh, 1u, __ATOMIC_RELAXED, __HIP_MEMORY_SCOPE_AGENT);
      const unsigned t = __builtin_amdgcn_readfirstlane(traw) - (call - 1) * (64u + cx);
      if (t < 64u) {
        const int j_ = t >> 4, d_ = (r + 1 + ((t >> 1) & 7)) & 7, nt_ = xcd * 2 + (t & 1);
        const unsigned* lf = land + (lane >> 3) * 256 + ((lane & 4) ? 128 + nt_ * 4 : (d_ * 4 + j_) * 4) + (lane & 3);
        const unsigned long long t0 = __builtin_amdgcn_s_memrealtime();
        while (__ballot(ld_agent(lf) < call) != 0ull) {
          if (ld_agent(tmo) == call || __builtin_amdgcn_s_memrealtime() - t0 > DEADLINE_TICKS) {
            if (lane == 0) { __hip_atomic_store(tmo, call, __ATOMIC_RELAXED, __HIP_MEMORY_SCOPE_AGENT); s_ok = 0; }
            break;
          }
          __builtin_amdgcn_s_sleep(2);
        }
        __builtin_amdgcn_fence(__ATOMIC_ACQUIRE, "agent");
        WAIT_VM0();
      }
      if (lane == 0) s_tile = t;
    }
    __syncthreads();
    const unsigned t = __builtin_amdgcn_readfirstlane(s_tile);
    if (t >= 64u) break;
    const int j = t >> 4, dd = (t >> 1) & 7, nt = xcd * 2 + (t & 1);
    const int d = (r + 1 + dd) & 7;
    const int mt = d * 4 + j;
    TR();
    f32x4 acc[4][4];
#pragma unroll
    for (int i = 0; i < 4; ++i)
#pragma unroll
      for (int k = 0; k < 4; ++k) acc[i][k] = (f32x4){0.f, 0.f, 0.f, 0.f};

    auto issue = [&](int s) {
      const int src = (r + (s >> 2)) & 7, kb = s & 3;
      const unsigned char* wb = (src == r) ? ws + OFF_WPUB + (size_t)r * MiB : ws + OFF_WLAND + (size_t)src * MiB;
      const unsigned char* xb = (src == r) ? ws + OFF_XPUB : ws + OFF_XLAND + (size_t)src * 2 * MiB;
      unsigned char* lb = lds + (s & 3) * 32768;
      const unsigned char* wsrc = wb + (size_t)(nt * 4 + kb) * 16384 + wid * 4096 + lane * 16;
      const unsigned char* xsrc = xb + (size_t)(mt * 4 + kb) * 16384 + wid * 4096 + lane * 16;
#pragma unroll
      for (int i = 0; i < 4; ++i)
        __builtin_amdgcn_global_load_lds((const unsigned*)(wsrc + i * 1024), (unsigned*)(lb + wid * 4096 + i * 1024), 16, 0, 0);
#pragma unroll
      for (int i = 0; i < 4; ++i)
        __builtin_amdgcn_global_load_lds((const unsigned*)(xsrc + i * 1024), (unsigned*)(lb + 16384 + wid * 4096 + i * 1024), 16, 0, 0);
    };
    auto rdfrag = [&](int s, i32x8* wf, i32x8* xf) {
      const unsigned char* lb = lds + (s & 3) * 32768 + lane * 16;
#pragma unroll
      for (int i = 0; i < 4; ++i) {
        wf[i] = ldfrag(lb + (wn * 4 + i) * 2048);
        xf[i] = ldfrag(lb + 16384 + (wm * 4 + i) * 2048);
      }
    };
    auto mma = [&](const i32x8* wf, const i32x8* xf) {
#pragma unroll
      for (int i = 0; i < 4; ++i)
#pragma unroll
        for (int k = 0; k < 4; ++k)
          acc[i][k] = __builtin_amdgcn_mfma_scale_f32_16x16x128_f8f6f4(wf[i], xf[k], acc[i][k], 0, 0, 0, 0x7F, 0, 0x7F);
    };
#define RAW_BAR() do { asm volatile("s_waitcnt lgkmcnt(0)" ::: "memory"); __builtin_amdgcn_s_barrier(); } while (0)
    i32x8 wfA[4], xfA[4], wfB[4], xfB[4];
    issue(0); issue(1); issue(2); issue(3);
    asm volatile("s_waitcnt vmcnt(24)" ::: "memory");
    RAW_BAR();
    rdfrag(0, wfA, xfA);
    for (int s = 0; s < 28; s += 2) {
      asm volatile("s_waitcnt vmcnt(16)" ::: "memory");
      RAW_BAR();
      issue(s + 4); rdfrag(s + 1, wfB, xfB); mma(wfA, xfA);
      asm volatile("s_waitcnt vmcnt(16)" ::: "memory");
      RAW_BAR();
      issue(s + 5); rdfrag(s + 2, wfA, xfA); mma(wfB, xfB);
    }
    asm volatile("s_waitcnt vmcnt(16)" ::: "memory");
    RAW_BAR();
    rdfrag(29, wfB, xfB); mma(wfA, xfA);
    asm volatile("s_waitcnt vmcnt(8)" ::: "memory");
    RAW_BAR();
    rdfrag(30, wfA, xfA); mma(wfB, xfB);
    asm volatile("s_waitcnt vmcnt(0)" ::: "memory");
    RAW_BAR();
    rdfrag(31, wfB, xfB); mma(wfA, xfA);
    mma(wfB, xfB);
    TR();

    const bool ok = s_ok != 0;
    if (d == r) {
      float* ob = p.out + (size_t)(j * 128) * NN + (size_t)r * NS + nt * 128;
#pragma unroll
      for (int i = 0; i < 4; ++i)
#pragma unroll
        for (int k = 0; k < 4; ++k) {
          f32x4 v = acc[i][k] * scale;
          if (!ok) v = (f32x4){__builtin_nanf(""), __builtin_nanf(""), __builtin_nanf(""), __builtin_nanf("")};
          const int m = (wm * 4 + k) * 16 + fc, n = (wn * 4 + i) * 16 + fq * 4;
          *(f32x4*)(ob + (size_t)m * NN + n) = v;
        }
      RAW_BAR();
    } else {
      const __amdgpu_buffer_rsrc_t rs_loc = mkrsrc(ws);
      const unsigned ob = (unsigned)(OFF_OUT + ((size_t)(d * 4 + j) * 16 + nt) * 65536);
#pragma unroll
      for (int i = 0; i < 4; ++i)
#pragma unroll
        for (int k = 0; k < 4; ++k) {
          f32x4 v = acc[i][k] * scale;
          if (!ok) v = (f32x4){__builtin_nanf(""), __builtin_nanf(""), __builtin_nanf(""), __builtin_nanf("")};
          const int m = (wm * 4 + k) * 16 + fc, n = (wn * 4 + i) * 16 + fq * 4;
          __builtin_amdgcn_raw_buffer_store_b128(__builtin_bit_cast(u32x4, v), rs_loc, ob + (unsigned)(m * 128 + n) * 4u, 0, AUX_SYS);
        }
      WAIT_VM0();
      RAW_BAR();
      if (tid == 0) {
        unsigned* f = (unsigned*)(p.ws[d] + OFF_FLAGS) + r * 128 + FLAG_O + j * 16 + nt;
        __hip_atomic_store(f, call, __ATOMIC_RELAXED, __HIP_MEMORY_SCOPE_SYSTEM);
      }
    }
    TR();
  }
}

}

extern "C" void kernel_launch(int rank, int nranks, ncclComm_t comm,
                              hipStream_t stream,
                              void* const* d_in, const int* in_sizes, int n_in,
                              void* d_out, int out_size,
                              const int* in_dtypes, int out_dtype,
                              void* const* d_out_all,
                              void* const* d_ws_all, size_t ws_bytes) {
  if (nranks != NR || ws_bytes < WS_NEED || rank < 0 || rank >= NR) {
    fprintf(stderr, "unexpected configuration\n");
    return;
  }
  Params p;
  p.x = (const unsigned char*)d_in[0];
  p.w = (const unsigned char*)d_in[1];
  p.sx = (const float*)d_in[2];
  p.sw = (const float*)d_in[3];
  p.out = (float*)d_out;
  for (int i = 0; i < NR; ++i) p.ws[i] = (unsigned char*)d_ws_all[i];
  p.rank = rank;
  hipLaunchKernelGGL(fused_kernel, dim3(GRID), dim3(NTHR), 0, stream, p);
  HIP_CHECK(hipGetLastError());
}
```

```cpp
#include <hip/hip_runtime.h>
#include <rccl/rccl.h>
#include <cstdio>
#include <cstdint>

#define HIP_CHECK(expr) do { hipError_t _e = (expr); if (_e != hipSuccess) { \
    fprintf(stderr, "HIP error %s at %s:%d\n", hipGetErrorString(_e), __FILE__, __LINE__); \
    return; } } while (0)

namespace {

typedef __attribute__((ext_vector_type(4))) unsigned u32x4;
typedef __attribute__((ext_vector_type(8))) int i32x8;
typedef __attribute__((ext_vector_type(4))) float f32x4;

constexpr int NR = 8;
constexpr int NN = 16384;
constexpr int KS = 512;
constexpr int NS = 2048;
#define GPULL 5
constexpr int G = GPULL;
constexpr int NPULL = (NR - 1) * G;
constexpr int GRID = 256;
constexpr int NTHR = 256;

constexpr size_t MiB = 1ull << 20;
constexpr size_t OFF_CALLCNT = 0;
constexpr size_t OFF_FLAGS = 4096;
constexpr size_t OFF_LAND = 16384;
constexpr size_t OFF_QUEUE = 10240;
constexpr size_t OFF_TICKET = 11264;
constexpr size_t OFF_TMO = 12288;
constexpr size_t OFF_TRACE = 65536;
constexpr size_t OFF_XPUB = 1 * MiB;
constexpr size_t OFF_WPUB = 8 * MiB;
constexpr size_t OFF_XLAND = 32 * MiB;
constexpr size_t OFF_WLAND = 48 * MiB;
constexpr size_t OFF_OUT = 64 * MiB;
constexpr size_t WS_NEED = 96 * MiB;

constexpr int FLAG_X0 = 0, FLAG_W = 32, FLAG_X = 48, FLAG_O = 72;
constexpr int FSTR = 192;
constexpr unsigned NBIG = 424;
constexpr unsigned NPIECE = NBIG + (448 - NBIG) * 4;
constexpr unsigned long long DEADLINE_TICKS = 50000000ull;

struct Params {
  const unsigned char* x;
  const unsigned char* w;
  const float* sx;
  const float* sw;
  float* out;
  unsigned char* ws[NR];
  int rank;
};

#define WAIT_VM0() asm volatile("s_waitcnt vmcnt(0)" ::: "memory")
#define TR() do {} while (0)
#define TR2(t) do {} while (0)

constexpr int AUX_SYS = 17;
constexpr int AUX_AGENT = 16;

__device__ __forceinline__ unsigned ld_sys(const unsigned* p) { return __hip_atomic_load(p, __ATOMIC_RELAXED, __HIP_MEMORY_SCOPE_SYSTEM); }
__device__ __forceinline__ unsigned ld_agent(const unsigned* p) { return __hip_atomic_load(p, __ATOMIC_RELAXED, __HIP_MEMORY_SCOPE_AGENT); }

__device__ __forceinline__ __amdgpu_buffer_rsrc_t mkrsrc(const void* base) {
  unsigned long long a = (unsigned long long)base;
  unsigned lo = __builtin_amdgcn_readfirstlane((unsigned)a), hi = __builtin_amdgcn_readfirstlane((unsigned)(a >> 32));
  void* pb = (void*)(((unsigned long long)hi << 32) | lo);
  return __builtin_amdgcn_make_buffer_rsrc(pb, 0, 0x7fffffff, 0x00020000);
}

__device__ __forceinline__ bool wait_ge(const unsigned* p, unsigned target, unsigned* tmo, unsigned call) {
  unsigned long long t0 = __builtin_amdgcn_s_memrealtime();
  while (ld_sys(p) < target) {
    if (ld_agent(tmo) == call || __builtin_amdgcn_s_memrealtime() - t0 > DEADLINE_TICKS) {
      __hip_atomic_store(tmo, call, __ATOMIC_RELAXED, __HIP_MEMORY_SCOPE_AGENT);
      return false;
    }
    __builtin_amdgcn_s_sleep(2);
  }
  return true;
}

__device__ __forceinline__ void transpose4(unsigned r0, unsigned r1, unsigned r2, unsigned r3,
                                           unsigned& c0, unsigned& c1, unsigned& c2, unsigned& c3) {
  unsigned t0 = __builtin_amdgcn_perm(r1, r0, 0x05010400u);
  unsigned t1 = __builtin_amdgcn_perm(r1, r0, 0x07030602u);
  unsigned t2 = __builtin_amdgcn_perm(r3, r2, 0x05010400u);
  unsigned t3 = __builtin_amdgcn_perm(r3, r2, 0x07030602u);
  c0 = __builtin_amdgcn_perm(t2, t0, 0x05040100u);
  c1 = __builtin_amdgcn_perm(t2, t0, 0x07060302u);
  c2 = __builtin_amdgcn_perm(t3, t1, 0x05040100u);
  c3 = __builtin_amdgcn_perm(t3, t1, 0x07060302u);
}

__device__ __forceinline__ i32x8 ldfrag(const unsigned char* p) {
  const u32x4 a0 = *(const u32x4*)(p);
  const u32x4 a1 = *(const u32x4*)(p + 1024);
  return (i32x8){(int)a0[0], (int)a0[1], (int)a0[2], (int)a0[3], (int)a1[0], (int)a1[1], (int)a1[2], (int)a1[3]};
}

__global__ void __launch_bounds__(NTHR, 1) fused_kernel(Params p) {
  __shared__ __attribute__((aligned(16))) unsigned char lds[131072 + 64];
  volatile unsigned* sctl = (volatile unsigned*)(lds + 131072);
#define s_call sctl[0]
#define s_ok sctl[1]
#define s_tile sctl[2]
  const int tid = threadIdx.x, lane = tid & 63;
  const int wid = __builtin_amdgcn_readfirstlane(tid >> 6);
  const int b = blockIdx.x;
  const int r = p.rank;
  unsigned char* ws = p.ws[r];
  unsigned* tmo = (unsigned*)(ws + OFF_TMO);
  unsigned* land = (unsigned*)(ws + OFF_LAND);
  int trn = 0; (void)trn;
  TR();

  if (b < NPULL) {
    const int pi = b % (NR - 1);
    const unsigned src = (unsigned)(r + 1 + pi) % NR;
    if (src >= (unsigned)NR) return;
    const unsigned char* rws = p.ws[src];
    const unsigned* fl = (const unsigned*)(ws + OFF_FLAGS) + src * FSTR;
    unsigned* tk = (unsigned*)(ws + OFF_TICKET) + src * 32;
    int zopq = 0; asm volatile("" : "+v"(zopq));
    tk += zopq;
    const unsigned* f2p = lane == 63 ? tmo : fl + 128 + lane;
    unsigned cprev = 0, traw = 0;
    unsigned* cc = (unsigned*)(ws + OFF_CALLCNT) + b;
    if (tid == 0) cprev = ld_agent(cc);
    if (lane == 0) traw = __hip_atomic_fetch_add(tk, 1u, __ATOMIC_RELAXED, __HIP_MEMORY_SCOPE_AGENT);
    unsigned f0 = ld_sys(fl + lane), f1 = ld_sys(fl + 64 + lane), f2 = ld_sys(f2p);
    if (tid == 0) {
      __hip_atomic_store(cc, cprev + 1, __ATOMIC_RELAXED, __HIP_MEMORY_SCOPE_AGENT);
      s_call = cprev + 1;
    }
    __syncthreads();
    const unsigned call = __builtin_amdgcn_readfirstlane(s_call);
    const __amdgpu_buffer_rsrc_t rs_rem = mkrsrc(rws);
    const __amdgpu_buffer_rsrc_t rs_loc = mkrsrc(ws);
    const __amdgpu_buffer_rsrc_t rs_out = mkrsrc(p.out);
    bool ok = true;
    unsigned* pend = nullptr;
    const unsigned tbase = (call - 1) * (NPIECE + 4u * G);
    const unsigned xpub = (unsigned)OFF_XPUB + (call & 1) * (unsigned)(2 * MiB), wpub = (unsigned)OFF_WPUB + (call & 1) * (unsigned)(8 * MiB);
    unsigned tn; unsigned long long rdy0, rdy1, rdy2;
#define NEXT_STATE() do { tn = __builtin_amdgcn_readfirstlane(traw) - tbase; rdy0 = __ballot(f0 >= call); rdy1 = __ballot(f1 >= call); \
      rdy2 = __ballot(lane != 63 && f2 >= call); if (__builtin_amdgcn_readlane(f2, 63) == call) ok = false; } while (0)
    NEXT_STATE();
    for (;;) {
      const unsigned t = tn;
      if (t >= NPIECE) break;
      if (lane == 0) traw = __hip_atomic_fetch_add(tk, 1u, __ATOMIC_RELAXED, __HIP_MEMORY_SCOPE_AGENT);
      unsigned fid, so, dof; unsigned* lc = nullptr; int b0 = 0, b1 = 4;
      if (t < 32u || (t >= 96u && t < 192u)) {
        const unsigned v = t < 32u ? t : t - 96u + 32u;
        const unsigned j = v >> 5, mt = ((v & 31) >> 2) * 4 + j;
        const unsigned o = mt * 65536u + (v & 3) * 16384u;
        fid = j == 0 ? FLAG_X0 + v : FLAG_X + (j - 1) * 8 + (mt >> 2); so = xpub + o; dof = (unsigned)(OFF_XLAND + (size_t)src * 2 * MiB) + o; lc = land + src * 256 + mt * 4 + (v & 3);
      } else if (t < 96u) {
        const unsigned u = t - 32u;
        fid = FLAG_W + (u >> 4) * 4 + (u & 3);
        so = wpub + (unsigned)r * (unsigned)MiB + u * 16384u; dof = (unsigned)(OFF_WLAND + (size_t)src * MiB) + u * 16384u; lc = land + src * 256 + 128 + u;
      } else {
        unsigned pc = t;
        if (t >= NBIG) { pc = NBIG + ((t - NBIG) >> 2); b0 = (t - NBIG) & 3; b1 = b0 + 1; }
        const unsigned v = pc - 192u, j = v >> 6, nt = (v >> 2) & 15, qt = v & 3;
        fid = FLAG_O + j * 16 + nt;
        so = (unsigned)(OFF_OUT + ((size_t)(r * 4 + j) * 16 + nt) * 65536) + qt * 16384u;
        dof = (unsigned)(((size_t)(j * 128 + qt * 32) * NN + (size_t)src * NS + nt * 128) * 4);
      }
      if (!(((fid < 64u ? rdy0 >> fid : fid < 128u ? rdy1 >> (fid - 64u) : rdy2 >> (fid - 128u)) & 1ull))) {
        if (pend) { WAIT_VM0(); if (lane == 0) __hip_atomic_store(pend, call, __ATOMIC_RELAXED, __HIP_MEMORY_SCOPE_AGENT); pend = nullptr; }
        ok = wait_ge(fl + fid, call, tmo, call) && ok;
      }
      f0 = ld_sys(fl + lane); f1 = ld_sys(fl + 64 + lane); f2 = ld_sys(f2p);
      if (t == 0u || t == 32u || t == 96u || t == 128u || t == 160u || t == 192u || t == 256u || t == 320u || t == 384u) TR2(t);
      if (lc) {
#pragma unroll
        for (int bt = 0; bt < 4; ++bt) {
          const unsigned o = (unsigned)bt * 4096u + lane * 16u;
          u32x4 v[4];
#pragma unroll
          for (int u = 0; u < 4; ++u) v[u] = __builtin_amdgcn_raw_buffer_load_b128(rs_rem, so + o + u * 1024u, 0, AUX_SYS);
          WAIT_VM0();
          if (bt == 3) NEXT_STATE();
          if (pend) { if (lane == 0) __hip_atomic_store(pend, call, __ATOMIC_RELAXED, __HIP_MEMORY_SCOPE_AGENT); pend = nullptr; }
#pragma unroll
          for (int u = 0; u < 4; ++u) __builtin_amdgcn_raw_buffer_store_b128(v[u], rs_loc, dof + o + u * 1024u, 0, AUX_AGENT);
        }
        pend = lc;
      } else {
        for (int bt = b0; bt < b1; ++bt) {
          u32x4 v[4];
          if (ok) {
#pragma unroll
            for (int u = 0; u < 4; ++u) v[u] = __builtin_amdgcn_raw_buffer_load_b128(rs_rem, so + bt * 4096u + u * 1024u + lane * 16u, 0, AUX_SYS);
          } else {
#pragma unroll
            for (int u = 0; u < 4; ++u) v[u] = (u32x4){0x7fc00000u, 0x7fc00000u, 0x7fc00000u, 0x7fc00000u};
          }
          WAIT_VM0();
          if (bt == b1 - 1) NEXT_STATE();
          if (pend) { if (lane == 0) __hip_atomic_store(pend, call, __ATOMIC_RELAXED, __HIP_MEMORY_SCOPE_AGENT); pend = nullptr; }
#pragma unroll
          for (int u = 0; u < 4; ++u) {
            const unsigned row = bt * 8 + u * 2 + (lane >> 5);
            __builtin_amdgcn_raw_buffer_store_b128(v[u], rs_out, dof + row * (unsigned)(NN * 4) + (lane & 31) * 16u, 0, 0);
          }
        }
      }
    }
    if (pend) { WAIT_VM0(); if (lane == 0) __hip_atomic_store(pend, call, __ATOMIC_RELAXED, __HIP_MEMORY_SCOPE_AGENT); }
    TR();
    return;
  }

  if (tid == 0) {
    unsigned* cc = (unsigned*)(ws + OFF_CALLCNT) + b;
    unsigned c = ld_agent(cc) + 1;
    __hip_atomic_store(cc, c, __ATOMIC_RELAXED, __HIP_MEMORY_SCOPE_AGENT);
    s_call = c; s_ok = 1;
  }
  const int gb = b - NPULL;
  if (gb < 184) {
    const __amdgpu_buffer_rsrc_t rs_loc = mkrsrc(ws);
    if (gb >= 56) {
      const int u = gb - 56;
      const int kb = u & 3, nq = u >> 2, d = nq >> 2, ntq = nq & 3;
      const int ng = tid & 31, kg = tid >> 5;
      const unsigned char* sp = p.w + (size_t)(kb * 128 + kg * 16) * NN + nq * 512 + ng * 16;
      u32x4 in[16];
#pragma unroll
      for (int i = 0; i < 16; ++i) in[i] = *(const u32x4*)(sp + (size_t)i * NN);
      u32x4 o[16];
#pragma unroll
      for (int v = 0; v < 4; ++v)
#pragma unroll
        for (int w = 0; w < 4; ++w) {
          unsigned c0, c1, c2, c3;
          transpose4(in[4 * v][w], in[4 * v + 1][w], in[4 * v + 2][w], in[4 * v + 3][w], c0, c1, c2, c3);
          o[4 * w][v] = c0; o[4 * w + 1][v] = c1; o[4 * w + 2][v] = c2; o[4 * w + 3][v] = c3;
        }
      const int ntl = ng >> 3, nb = ng & 7, h = kg >> 2, q = kg & 3;
      const int base16 = ntl * 1024 + nb * 128 + h * 64 + q * 16;
#pragma unroll
      for (int c = 0; c < 16; ++c) *(u32x4*)(lds + (size_t)((base16 + c) ^ nb) * 16) = o[c];
      __syncthreads();
      const unsigned dof = (unsigned)(OFF_WPUB + (size_t)d * MiB) + (s_call & 1) * (unsigned)(8 * MiB);
#pragma unroll
      for (int i = 0; i < 16; ++i) {
        const int L = i * NTHR + tid;
        u32x4 v = *(const u32x4*)(lds + (size_t)(L ^ ((L >> 7) & 7)) * 16);
        const int nt = ntq * 4 + (L >> 10);
        __builtin_amdgcn_raw_buffer_store_b128(v, rs_loc, dof + (unsigned)(nt * 4 + kb) * 16384u + (unsigned)(L & 1023) * 16u, 0, AUX_SYS);
      }
      WAIT_VM0();
      __syncthreads();
      if (tid < 4) {
        const unsigned call = s_call;
        if (d == r) {
          __hip_atomic_store(land + r * 256 + 128 + (ntq * 4 + tid) * 4 + kb, call, __ATOMIC_RELAXED, __HIP_MEMORY_SCOPE_AGENT);
        } else if (tid == 0) {
          unsigned* f = (unsigned*)(p.ws[d] + OFF_FLAGS) + r * FSTR + FLAG_W + ntq * 4 + kb;
          __hip_atomic_store(f, call, __ATOMIC_RELAXED, __HIP_MEMORY_SCOPE_SYSTEM);
        }
      }
    } else if (gb >= 32) {
      const int j = 1 + ((gb - 32) >> 3), mt = ((gb - 32) & 7) * 4 + j;
      u32x4 v[16];
#pragma unroll
      for (int i = 0; i < 16; ++i) {
        const int L = i * NTHR + tid;
        const int kb = L >> 10, mb = (L >> 7) & 7, h = (L >> 6) & 1, q = (L >> 4) & 3, c = L & 15;
        v[i] = *(const u32x4*)(p.x + (size_t)(mt * 128 + mb * 16 + c) * KS + kb * 128 + h * 64 + q * 16);
      }
      __syncthreads();
      const unsigned xpub = (unsigned)OFF_XPUB + (s_call & 1) * (unsigned)(2 * MiB);
#pragma unroll
      for (int i = 0; i < 16; ++i)
        __builtin_amdgcn_raw_buffer_store_b128(v[i], rs_loc, xpub + (unsigned)mt * 65536u + (unsigned)(i * NTHR + tid) * 16u, 0, AUX_SYS);
      WAIT_VM0();
      __syncthreads();
      if (tid < NR + 4) {
        const unsigned call = s_call;
        if (tid >= NR) {
          __hip_atomic_store(land + r * 256 + mt * 4 + (tid - NR), call, __ATOMIC_RELAXED, __HIP_MEMORY_SCOPE_AGENT);
        } else if (tid != r) {
          unsigned* f = (unsigned*)(p.ws[tid] + OFF_FLAGS) + r * FSTR + FLAG_X + (gb - 32);
          __hip_atomic_store(f, call, __ATOMIC_RELAXED, __HIP_MEMORY_SCOPE_SYSTEM);
        }
      }
    } else {
      const int mt = (gb >> 2) * 4, kb = gb & 3;
      u32x4 v[4];
#pragma unroll
      for (int i = 0; i < 4; ++i) {
        const int L = i * NTHR + tid;
        const int mb = (L >> 7) & 7, h = (L >> 6) & 1, q = (L >> 4) & 3, c = L & 15;
        v[i] = *(const u32x4*)(p.x + (size_t)(mt * 128 + mb * 16 + c) * KS + kb * 128 + h * 64 + q * 16);
      }
      __syncthreads();
      const unsigned xpub = (unsigned)OFF_XPUB + (s_call & 1) * (unsigned)(2 * MiB);
#pragma unroll
      for (int i = 0; i < 4; ++i)
        __builtin_amdgcn_raw_buffer_store_b128(v[i], rs_loc, xpub + (unsigned)(mt * 4 + kb) * 16384u + (unsigned)(i * NTHR + tid) * 16u, 0, AUX_SYS);
      WAIT_VM0();
      __syncthreads();
      if (tid < NR) {
        const unsigned call = s_call;
        if (tid == r) {
          __hip_atomic_store(land + r * 256 + mt * 4 + kb, call, __ATOMIC_RELAXED, __HIP_MEMORY_SCOPE_AGENT);
        } else {
          unsigned* f = (unsigned*)(p.ws[tid] + OFF_FLAGS) + r * FSTR + FLAG_X0 + gb;
          __hip_atomic_store(f, call, __ATOMIC_RELAXED, __HIP_MEMORY_SCOPE_SYSTEM);
        }
      }
    }
  }
  __syncthreads();
  const unsigned call = __builtin_amdgcn_readfirstlane(s_call);
  TR();

  const float scale = p.sx[0] * p.sw[0];
  const int xcd = b & 7;
  const int first = NPULL + ((xcd - NPULL) & 7);
  const unsigned cx = (unsigned)(GRID - first + 7) >> 3;
  unsigned* qh = (unsigned*)(ws + OFF_QUEUE) + xcd * 32;
  const int wn = wid >> 1, wm = wid & 1;
  const int fq = lane >> 4, fc = lane & 15;
  for (;;) {
    if (wid == 0) {
      unsigned traw = 0;
      if (lane == 0) traw = __hip_atomic_fetch_add(qh, 1u, __ATOMIC_RELAXED, __HIP_MEMORY_SCOPE_AGENT);
      const unsigned t = __builtin_amdgcn_readfirstlane(traw) - (call - 1) * (64u + cx);
      if (t < 64u) {
        const int j_ = t >> 4, d_ = (r + 1 + ((t >> 1) & 7)) & 7, nt_ = xcd * 2 + (t & 1);
        const unsigned* lf = land + (lane >> 3) * 256 + ((lane & 4) ? 128 + nt_ * 4 : (d_ * 4 + j_) * 4) + (lane & 3);
        const unsigned long long t0 = __builtin_amdgcn_s_memrealtime();
        for (;;) {
          const unsigned lv = ld_agent(lf), tv = ld_agent(tmo);
          if (tv == call) { if (lane == 0) s_ok = 0; break; }
          if (__ballot(lv < call) == 0ull) break;
          if (__builtin_amdgcn_s_memrealtime() - t0 > DEADLINE_TICKS) {
            if (lane == 0) { __hip_atomic_store(tmo, call, __ATOMIC_RELAXED, __HIP_MEMORY_SCOPE_AGENT); s_ok = 0; }
            break;
          }
          __builtin_amdgcn_s_sleep(2);
        }
        __builtin_amdgcn_fence(__ATOMIC_ACQUIRE, "agent");
        WAIT_VM0();
      }
      if (lane == 0) s_tile = t;
    }
    __syncthreads();
    const unsigned t = __builtin_amdgcn_readfirstlane(s_tile);
    if (t >= 64u) break;
    const int j = t >> 4, dd = (t >> 1) & 7, nt = xcd * 2 + (t & 1);
    const int d = (r + 1 + dd) & 7;
    const int mt = d * 4 + j;
    TR();
    f32x4 acc[4][4];
#pragma unroll
    for (int i = 0; i < 4; ++i)
#pragma unroll
      for (int k = 0; k < 4; ++k) acc[i][k] = (f32x4){0.f, 0.f, 0.f, 0.f};

    auto issue = [&](int s) {
      const int src = (r + (s >> 2)) & 7, kb = s & 3;
      const unsigned char* wb = (src == r) ? ws + OFF_WPUB + (size_t)(r + (call & 1) * 8) * MiB : ws + OFF_WLAND + (size_t)src * MiB;
      const unsigned char* xb = (src == r) ? ws + OFF_XPUB + (size_t)(call & 1) * 2 * MiB : ws + OFF_XLAND + (size_t)src * 2 * MiB;
      unsigned char* lb = lds + (s & 3) * 32768;
      const unsigned char* wsrc = wb + (size_t)(nt * 4 + kb) * 16384 + wid * 4096 + lane * 16;
      const unsigned char* xsrc = xb + (size_t)(mt * 4 + kb) * 16384 + wid * 4096 + lane * 16;
#pragma unroll
      for (int i = 0; i < 4; ++i)
        __builtin_amdgcn_global_load_lds((const unsigned*)(wsrc + i * 1024), (unsigned*)(lb + wid * 4096 + i * 1024), 16, 0, 0);
#pragma unroll
      for (int i = 0; i < 4; ++i)
        __builtin_amdgcn_global_load_lds((const unsigned*)(xsrc + i * 1024), (unsigned*)(lb + 16384 + wid * 4096 + i * 1024), 16, 0, 0);
    };
    auto rdfrag = [&](int s, i32x8* wf, i32x8* xf) {
      const unsigned char* lb = lds + (s & 3) * 32768 + lane * 16;
#pragma unroll
      for (int i = 0; i < 4; ++i) {
        wf[i] = ldfrag(lb + (wn * 4 + i) * 2048);
        xf[i] = ldfrag(lb + 16384 + (wm * 4 + i) * 2048);
      }
    };
    auto mma = [&](const i32x8* wf, const i32x8* xf) {
#pragma unroll
      for (int i = 0; i < 4; ++i)
#pragma unroll
        for (int k = 0; k < 4; ++k)
          acc[i][k] = __builtin_amdgcn_mfma_scale_f32_16x16x128_f8f6f4(wf[i], xf[k], acc[i][k], 0, 0, 0, 0x7F, 0, 0x7F);
    };
#define RAW_BAR() do { asm volatile("s_waitcnt lgkmcnt(0)" ::: "memory"); __builtin_amdgcn_s_barrier(); } while (0)
    i32x8 wfA[4], xfA[4], wfB[4], xfB[4];
    issue(0); issue(1); issue(2); issue(3);
    asm volatile("s_waitcnt vmcnt(24)" ::: "memory");
    RAW_BAR();
    rdfrag(0, wfA, xfA);
    for (int s = 0; s < 28; s += 2) {
      asm volatile("s_waitcnt vmcnt(16)" ::: "memory");
      RAW_BAR();
      issue(s + 4); rdfrag(s + 1, wfB, xfB); mma(wfA, xfA);
      asm volatile("s_waitcnt vmcnt(16)" ::: "memory");
      RAW_BAR();
      issue(s + 5); rdfrag(s + 2, wfA, xfA); mma(wfB, xfB);
    }
    asm volatile("s_waitcnt vmcnt(16)" ::: "memory");
    RAW_BAR();
    rdfrag(29, wfB, xfB); mma(wfA, xfA);
    asm volatile("s_waitcnt vmcnt(8)" ::: "memory");
    RAW_BAR();
    rdfrag(30, wfA, xfA); mma(wfB, xfB);
    asm volatile("s_waitcnt vmcnt(0)" ::: "memory");
    RAW_BAR();
    rdfrag(31, wfB, xfB); mma(wfA, xfA);
    mma(wfB, xfB);
    TR();

    const bool ok = s_ok != 0;
    if (d == r) {
      float* ob = p.out + (size_t)(j * 128) * NN + (size_t)r * NS + nt * 128;
#pragma unroll
      for (int i = 0; i < 4; ++i)
#pragma unroll
        for (int k = 0; k < 4; ++k) {
          f32x4 v = acc[i][k] * scale;
          if (!ok) v = (f32x4){__builtin_nanf(""), __builtin_nanf(""), __builtin_nanf(""), __builtin_nanf("")};
          const int m = (wm * 4 + k) * 16 + fc, n = (wn * 4 + i) * 16 + fq * 4;
          *(f32x4*)(ob + (size_t)m * NN + n) = v;
        }
      RAW_BAR();
    } else {
      const __amdgpu_buffer_rsrc_t rs_loc = mkrsrc(ws);
      const unsigned ob = (unsigned)(OFF_OUT + ((size_t)(d * 4 + j) * 16 + nt) * 65536);
#pragma unroll
      for (int i = 0; i < 4; ++i)
#pragma unroll
        for (int k = 0; k < 4; ++k) {
          f32x4 v = acc[i][k] * scale;
          if (!ok) v = (f32x4){__builtin_nanf(""), __builtin_nanf(""), __builtin_nanf(""), __builtin_nanf("")};
          const int m = (wm * 4 + k) * 16 + fc, n = (wn * 4 + i) * 16 + fq * 4;
          __builtin_amdgcn_raw_buffer_store_b128(__builtin_bit_cast(u32x4, v), rs_loc, ob + (unsigned)(m * 128 + n) * 4u, 0, AUX_SYS);
        }
      WAIT_VM0();
      RAW_BAR();
      if (tid == 0) {
        unsigned* f = (unsigned*)(p.ws[d] + OFF_FLAGS) + r * FSTR + FLAG_O + j * 16 + nt;
        __hip_atomic_store(f, call, __ATOMIC_RELAXED, __HIP_MEMORY_SCOPE_SYSTEM);
      }
    }
    TR();
  }
}

}

extern "C" void kernel_launch(int rank, int nranks, ncclComm_t comm,
                              hipStream_t stream,
                              void* const* d_in, const int* in_sizes, int n_in,
                              void* d_out, int out_size,
                              const int* in_dtypes, int out_dtype,
                              void* const* d_out_all,
                              void* const* d_ws_all, size_t ws_bytes) {
  if (nranks != NR || ws_bytes < WS_NEED || rank < 0 || rank >= NR) {
    fprintf(stderr, "unexpected configuration\n");
    return;
  }
  Params p;
  p.x = (const unsigned char*)d_in[0];
  p.w = (const unsigned char*)d_in[1];
  p.sx = (const float*)d_in[2];
  p.sw = (const float*)d_in[3];
  p.out = (float*)d_out;
  for (int i = 0; i < NR; ++i) p.ws[i] = (unsigned char*)d_ws_all[i];
  p.rank = rank;
  hipLaunchKernelGGL(fused_kernel, dim3(GRID), dim3(NTHR), 0, stream, p);
  HIP_CHECK(hipGetLastError());
}
```

```cpp
#include <hip/hip_runtime.h>
#include <rccl/rccl.h>
#include <cstdio>
#include <cstdint>

#define HIP_CHECK(expr) do { hipError_t _e = (expr); if (_e != hipSuccess) { \
    fprintf(stderr, "HIP error %s at %s:%d\n", hipGetErrorString(_e), __FILE__, __LINE__); \
    return; } } while (0)

namespace {

typedef __attribute__((ext_vector_type(4))) unsigned u32x4;
typedef __attribute__((ext_vector_type(8))) int i32x8;
typedef __attribute__((ext_vector_type(4))) float f32x4;

constexpr int NR = 8;
constexpr int NN = 16384;
constexpr int KS = 512;
constexpr int NS = 2048;
#define OUTAUX 16
#define GPULL 6
constexpr int G = GPULL;
#define NWV 18
constexpr int NPULL = (NR - 1) * G;
constexpr int GRID = 256;
constexpr int NTHR = 256;

constexpr size_t MiB = 1ull << 20;
constexpr size_t OFF_CALLCNT = 0;
constexpr size_t OFF_FLAGS = 4096;
constexpr size_t OFF_LAND = 16384;
constexpr size_t OFF_QUEUE = 10240;
constexpr size_t OFF_TICKET = 11264;
constexpr size_t OFF_TMO = 12288;
constexpr size_t OFF_TRACE = 65536;
constexpr size_t OFF_XPUB = 1 * MiB;
constexpr size_t OFF_WPUB = 8 * MiB;
constexpr size_t OFF_XLAND = 32 * MiB;
constexpr size_t OFF_WLAND = 48 * MiB;
constexpr size_t OFF_OUT = 64 * MiB;
constexpr size_t WS_NEED = 96 * MiB;

constexpr int FLAG_X0 = 0, FLAG_W = 32, FLAG_X = 48, FLAG_O = 72;
constexpr int FSTR = 192;
constexpr unsigned NBIG = 424;
constexpr unsigned NPIECE = NBIG + (448 - NBIG) * 4;
constexpr unsigned long long DEADLINE_TICKS = 50000000ull;

struct Params {
  const unsigned char* x;
  const unsigned char* w;
  const float* sx;
  const float* sw;
  float* out;
  unsigned char* ws[NR];
  int rank;
};

#define WAIT_VM0() asm volatile("s_waitcnt vmcnt(0)" ::: "memory")
#define TR() do {} while (0)
#define TR2(t) do {} while (0)

constexpr int AUX_SYS = 17;
constexpr int AUX_AGENT = 16;

__device__ __forceinline__ unsigned ld_sys(const unsigned* p) { return __hip_atomic_load(p, __ATOMIC_RELAXED, __HIP_MEMORY_SCOPE_SYSTEM); }
__device__ __forceinline__ unsigned ld_agent(const unsigned* p) { return __hip_atomic_load(p, __ATOMIC_RELAXED, __HIP_MEMORY_SCOPE_AGENT); }

__device__ __forceinline__ __amdgpu_buffer_rsrc_t mkrsrc(const void* base) {
  unsigned long long a = (unsigned long long)base;
  unsigned lo = __builtin_amdgcn_readfirstlane((unsigned)a), hi = __builtin_amdgcn_readfirstlane((unsigned)(a >> 32));
  void* pb = (void*)(((unsigned long long)hi << 32) | lo);
  return __builtin_amdgcn_make_buffer_rsrc(pb, 0, 0x7fffffff, 0x00020000);
}

__device__ __forceinline__ bool wait_ge(const unsigned* p, unsigned target, unsigned* tmo, unsigned call) {
  unsigned long long t0 = __builtin_amdgcn_s_memrealtime();
  while (ld_sys(p) < target) {
    if (ld_agent(tmo) == call || __builtin_amdgcn_s_memrealtime() - t0 > DEADLINE_TICKS) {
      __hip_atomic_store(tmo, call, __ATOMIC_RELAXED, __HIP_MEMORY_SCOPE_AGENT);
      return false;
    }
    __builtin_amdgcn_s_sleep(2);
  }
  return true;
}

__device__ __forceinline__ void transpose4(unsigned r0, unsigned r1, unsigned r2, unsigned r3,
                                           unsigned& c0, unsigned& c1, unsigned& c2, unsigned& c3) {
  unsigned t0 = __builtin_amdgcn_perm(r1, r0, 0x05010400u);
  unsigned t1 = __builtin_amdgcn_perm(r1, r0, 0x07030602u);
  unsigned t2 = __builtin_amdgcn_perm(r3, r2, 0x05010400u);
  unsigned t3 = __builtin_amdgcn_perm(r3, r2, 0x07030602u);
  c0 = __builtin_amdgcn_perm(t2, t0, 0x05040100u);
  c1 = __builtin_amdgcn_perm(t2, t0, 0x07060302u);
  c2 = __builtin_amdgcn_perm(t3, t1, 0x05040100u);
  c3 = __builtin_amdgcn_perm(t3, t1, 0x07060302u);
}

__device__ __forceinline__ i32x8 ldfrag(const unsigned char* p) {
  const u32x4 a0 = *(const u32x4*)(p);
  const u32x4 a1 = *(const u32x4*)(p + 1024);
  return (i32x8){(int)a0[0], (int)a0[1], (int)a0[2], (int)a0[3], (int)a1[0], (int)a1[1], (int)a1[2], (int)a1[3]};
}

__global__ void __launch_bounds__(NTHR, 1) fused_kernel(Params p) {
  __shared__ __attribute__((aligned(16))) unsigned char lds[131072 + 64];
  volatile unsigned* sctl = (volatile unsigned*)(lds + 131072);
#define s_call sctl[0]
#define s_ok sctl[1]
#define s_tile sctl[2]
  const int tid = threadIdx.x, lane = tid & 63;
  const int wid = __builtin_amdgcn_readfirstlane(tid >> 6);
  const int b = blockIdx.x;
  const int r = p.rank;
  unsigned char* ws = p.ws[r];
  unsigned* tmo = (unsigned*)(ws + OFF_TMO);
  unsigned* land = (unsigned*)(ws + OFF_LAND);
  int trn = 0; (void)trn;
  TR();

  if (b < NPULL) {
    const int pi = b % (NR - 1);
    const unsigned src = (unsigned)(r + 1 + pi) % NR;
    if (src >= (unsigned)NR) return;
    const unsigned char* rws = p.ws[src];
    const unsigned* fl = (const unsigned*)(ws + OFF_FLAGS) + src * FSTR;
    unsigned* tk = (unsigned*)(ws + OFF_TICKET) + src * 32;
    int zopq = 0; asm volatile("" : "+v"(zopq));
    tk += zopq;
    const unsigned* f2p = lane == 63 ? tmo : fl + 128 + lane;
    unsigned cprev = 0, traw = 0;
    unsigned* cc = (unsigned*)(ws + OFF_CALLCNT) + b;
    if (tid == 0) cprev = ld_agent(cc);
    const bool active = wid * G + b / (NR - 1) < (NWV);
    if (lane == 0 && active) traw = __hip_atomic_fetch_add(tk, 1u, __ATOMIC_RELAXED, __HIP_MEMORY_SCOPE_AGENT);
    unsigned f0 = ld_sys(fl + lane), f1 = ld_sys(fl + 64 + lane), f2 = ld_sys(f2p);
    if (tid == 0) {
      __hip_atomic_store(cc, cprev + 1, __ATOMIC_RELAXED, __HIP_MEMORY_SCOPE_AGENT);
      s_call = cprev + 1;
    }
    __syncthreads();
    const unsigned call = __builtin_amdgcn_readfirstlane(s_call);
    if (!active) return;
    const __amdgpu_buffer_rsrc_t rs_rem = mkrsrc(rws);
    const __amdgpu_buffer_rsrc_t rs_loc = mkrsrc(ws);
    const __amdgpu_buffer_rsrc_t rs_out = mkrsrc(p.out);
    bool ok = true;
    unsigned* pend = nullptr;
    const unsigned tbase = (call - 1) * (NPIECE + (unsigned)(NWV));
    const unsigned xpub = (unsigned)OFF_XPUB + (call & 1) * (unsigned)(2 * MiB), wpub = (unsigned)OFF_WPUB + (call & 1) * (unsigned)(8 * MiB);
    unsigned tn; unsigned long long rdy0, rdy1, rdy2;
#define NEXT_STATE() do { tn = __builtin_amdgcn_readfirstlane(traw) - tbase; rdy0 = __ballot(f0 >= call); rdy1 = __ballot(f1 >= call); \
      rdy2 = __ballot(lane != 63 && f2 >= call); if (__builtin_amdgcn_readlane(f2, 63) == call) ok = false; } while (0)
    NEXT_STATE();
    for (;;) {
      const unsigned t = tn;
      if (t >= NPIECE) break;
      if (lane == 0) traw = __hip_atomic_fetch_add(tk, 1u, __ATOMIC_RELAXED, __HIP_MEMORY_SCOPE_AGENT);
      unsigned fid, so, dof; unsigned* lc = nullptr; int b0 = 0, b1 = 4;
      if (t < 32u || (t >= 96u && t < 192u)) {
        const unsigned v = t < 32u ? t : t - 96u + 32u;
        const unsigned j = v >> 5, mt = ((v & 31) >> 2) * 4 + j;
        const unsigned o = mt * 65536u + (v & 3) * 16384u;
        fid = j == 0 ? FLAG_X0 + v : FLAG_X + (j - 1) * 8 + (mt >> 2); so = xpub + o; dof = (unsigned)(OFF_XLAND + (size_t)src * 2 * MiB) + o; lc = land + src * 256 + mt * 4 + (v & 3);
      } else if (t < 96u) {
        const unsigned u = t - 32u;
        fid = FLAG_W + (u >> 4) * 4 + (u & 3);
        so = wpub + (unsigned)r * (unsigned)MiB + u * 16384u; dof = (unsigned)(OFF_WLAND + (size_t)src * MiB) + u * 16384u; lc = land + src * 256 + 128 + u;
      } else {
        unsigned pc = t;
        if (t >= NBIG) { pc = NBIG + ((t - NBIG) >> 2); b0 = (t - NBIG) & 3; b1 = b0 + 1; }
        const unsigned v = pc - 192u, j = v >> 6, nt = (v >> 2) & 15, qt = v & 3;
        fid = FLAG_O + j * 16 + nt;
        so = (unsigned)(OFF_OUT + ((size_t)(r * 4 + j) * 16 + nt) * 65536) + qt * 16384u;
        dof = (unsigned)(((size_t)(j * 128 + qt * 32) * NN + (size_t)src * NS + nt * 128) * 4);
      }
      if (!(((fid < 64u ? rdy0 >> fid : fid < 128u ? rdy1 >> (fid - 64u) : rdy2 >> (fid - 128u)) & 1ull))) {
        if (pend) { WAIT_VM0(); if (lane == 0) __hip_atomic_store(pend, call, __ATOMIC_RELAXED, __HIP_MEMORY_SCOPE_AGENT); pend = nullptr; }
        ok = wait_ge(fl + fid, call, tmo, call) && ok;
      }
      f0 = ld_sys(fl + lane); f1 = ld_sys(fl + 64 + lane); f2 = ld_sys(f2p);
      if (t == 0u || t == 32u || t == 96u || t == 128u || t == 160u || t == 192u || t == 256u || t == 320u || t == 384u) TR2(t);
      if (lc) {
#pragma unroll
        for (int bt = 0; bt < 4; ++bt) {
          const unsigned o = (unsigned)bt * 4096u + lane * 16u;
          u32x4 v[4];
#pragma unroll
          for (int u = 0; u < 4; ++u) v[u] = __builtin_amdgcn_raw_buffer_load_b128(rs_rem, so + o + u * 1024u, 0, AUX_SYS);
          WAIT_VM0();
          if (bt == 3) NEXT_STATE();
          if (pend) { if (lane == 0) __hip_atomic_store(pend, call, __ATOMIC_RELAXED, __HIP_MEMORY_SCOPE_AGENT); pend = nullptr; }
#pragma unroll
          for (int u = 0; u < 4; ++u) __builtin_amdgcn_raw_buffer_store_b128(v[u], rs_loc, dof + o + u * 1024u, 0, AUX_AGENT);
        }
        pend = lc;
      } else {
        for (int bt = b0; bt < b1; ++bt) {
          u32x4 v[4];
          if (ok) {
#pragma unroll
            for (int u = 0; u < 4; ++u) v[u] = __builtin_amdgcn_raw_buffer_load_b128(rs_rem, so + bt * 4096u + u * 1024u + lane * 16u, 0, AUX_SYS);
          } else {
#pragma unroll
            for (int u = 0; u < 4; ++u) v[u] = (u32x4){0x7fc00000u, 0x7fc00000u, 0x7fc00000u, 0x7fc00000u};
          }
          WAIT_VM0();
          if (bt == b1 - 1) NEXT_STATE();
          if (pend) { if (lane == 0) __hip_atomic_store(pend, call, __ATOMIC_RELAXED, __HIP_MEMORY_SCOPE_AGENT); pend = nullptr; }
#pragma unroll
          for (int u = 0; u < 4; ++u) {
            const unsigned row = bt * 8 + u * 2 + (lane >> 5);
            __builtin_amdgcn_raw_buffer_store_b128(v[u], rs_out, dof + row * (unsigned)(NN * 4) + (lane & 31) * 16u, 0, OUTAUX);
          }
        }
      }
    }
    if (pend) { WAIT_VM0(); if (lane == 0) __hip_atomic_store(pend, call, __ATOMIC_RELAXED, __HIP_MEMORY_SCOPE_AGENT); }
    TR();
    return;
  }

  if (tid == 0) {
    unsigned* cc = (unsigned*)(ws + OFF_CALLCNT) + b;
    unsigned c = ld_agent(cc) + 1;
    __hip_atomic_store(cc, c, __ATOMIC_RELAXED, __HIP_MEMORY_SCOPE_AGENT);
    s_call = c; s_ok = 1;
  }
  const int gb = b - NPULL;
  if (gb < 184) {
    const __amdgpu_buffer_rsrc_t rs_loc = mkrsrc(ws);
    if (gb >= 56) {
      const int u = gb - 56;
      const int kb = u & 3, nq = u >> 2, d = nq >> 2, ntq = nq & 3;
      const int ng = tid & 31, kg = tid >> 5;
      const unsigned char* sp = p.w + (size_t)(kb * 128 + kg * 16) * NN + nq * 512 + ng * 16;
      u32x4 in[16];
#pragma unroll
      for (int i = 0; i < 16; ++i) in[i] = *(const u32x4*)(sp + (size_t)i * NN);
      u32x4 o[16];
#pragma unroll
      for (int v = 0; v < 4; ++v)
#pragma unroll
        for (int w = 0; w < 4; ++w) {
          unsigned c0, c1, c2, c3;
          transpose4(in[4 * v][w], in[4 * v + 1][w], in[4 * v + 2][w], in[4 * v + 3][w], c0, c1, c2, c3);
          o[4 * w][v] = c0; o[4 * w + 1][v] = c1; o[4 * w + 2][v] = c2; o[4 * w + 3][v] = c3;
        }
      const int ntl = ng >> 3, nb = ng & 7, h = kg >> 2, q = kg & 3;
      const int base16 = ntl * 1024 + nb * 128 + h * 64 + q * 16;
#pragma unroll
      for (int c = 0; c < 16; ++c) *(u32x4*)(lds + (size_t)((base16 + c) ^ nb) * 16) = o[c];
      __syncthreads();
      const unsigned dof = (unsigned)(OFF_WPUB + (size_t)d * MiB) + (s_call & 1) * (unsigned)(8 * MiB);
#pragma unroll
      for (int i = 0; i < 16; ++i) {
        const int L = i * NTHR + tid;
        u32x4 v = *(const u32x4*)(lds + (size_t)(L ^ ((L >> 7) & 7)) * 16);
        const int nt = ntq * 4 + (L >> 10);
        __builtin_amdgcn_raw_buffer_store_b128(v, rs_loc, dof + (unsigned)(nt * 4 + kb) * 16384u + (unsigned)(L & 1023) * 16u, 0, AUX_SYS);
      }
      WAIT_VM0();
      __syncthreads();
      if (tid < 4) {
        const unsigned call = s_call;
        if (d == r) {
          __hip_atomic_store(land + r * 256 + 128 + (ntq * 4 + tid) * 4 + kb, call, __ATOMIC_RELAXED, __HIP_MEMORY_SCOPE_AGENT);
        } else if (tid == 0) {
          unsigned* f = (unsigned*)(p.ws[d] + OFF_FLAGS) + r * FSTR + FLAG_W + ntq * 4 + kb;
          __hip_atomic_store(f, call, __ATOMIC_RELAXED, __HIP_MEMORY_SCOPE_SYSTEM);
        }
      }
    } else if (gb >= 32) {
      const int j = 1 + ((gb - 32) >> 3), mt = ((gb - 32) & 7) * 4 + j;
      u32x4 v[16];
#pragma unroll
      for (int i = 0; i < 16; ++i) {
        const int L = i * NTHR + tid;
        const int kb = L >> 10, mb = (L >> 7) & 7, h = (L >> 6) & 1, q = (L >> 4) & 3, c = L & 15;
        v[i] = *(const u32x4*)(p.x + (size_t)(mt * 128 + mb * 16 + c) * KS + kb * 128 + h * 64 + q * 16);
      }
      __syncthreads();
      const unsigned xpub = (unsigned)OFF_XPUB + (s_call & 1) * (unsigned)(2 * MiB);
#pragma unroll
      for (int i = 0; i < 16; ++i)
        __builtin_amdgcn_raw_buffer_store_b128(v[i], rs_loc, xpub + (unsigned)mt * 65536u + (unsigned)(i * NTHR + tid) * 16u, 0, AUX_SYS);
      WAIT_VM0();
      __syncthreads();
      if (tid < NR + 4) {
        const unsigned call = s_call;
        if (tid >= NR) {
          __hip_atomic_store(land + r * 256 + mt * 4 + (tid - NR), call, __ATOMIC_RELAXED, __HIP_MEMORY_SCOPE_AGENT);
        } else if (tid != r) {
          unsigned* f = (unsigned*)(p.ws[tid] + OFF_FLAGS) + r * FSTR + FLAG_X + (gb - 32);
          __hip_atomic_store(f, call, __ATOMIC_RELAXED, __HIP_MEMORY_SCOPE_SYSTEM);
        }
      }
    } else {
      const int mt = (gb >> 2) * 4, kb = gb & 3;
      u32x4 v[4];
#pragma unroll
      for (int i = 0; i < 4; ++i) {
        const int L = i * NTHR + tid;
        const int mb = (L >> 7) & 7, h = (L >> 6) & 1, q = (L >> 4) & 3, c = L & 15;
        v[i] = *(const u32x4*)(p.x + (size_t)(mt * 128 + mb * 16 + c) * KS + kb * 128 + h * 64 + q * 16);
      }
      __syncthreads();
      const unsigned xpub = (unsigned)OFF_XPUB + (s_call & 1) * (unsigned)(2 * MiB);
#pragma unroll
      for (int i = 0; i < 4; ++i)
        __builtin_amdgcn_raw_buffer_store_b128(v[i], rs_loc, xpub + (unsigned)(mt * 4 + kb) * 16384u + (unsigned)(i * NTHR + tid) * 16u, 0, AUX_SYS);
      WAIT_VM0();
      __syncthreads();
      if (tid < NR) {
        const unsigned call = s_call;
        if (tid == r) {
          __hip_atomic_store(land + r * 256 + mt * 4 + kb, call, __ATOMIC_RELAXED, __HIP_MEMORY_SCOPE_AGENT);
        } else {
          unsigned* f = (unsigned*)(p.ws[tid] + OFF_FLAGS) + r * FSTR + FLAG_X0 + gb;
          __hip_atomic_store(f, call, __ATOMIC_RELAXED, __HIP_MEMORY_SCOPE_SYSTEM);
        }
      }
    }
  }
  __syncthreads();
  const unsigned call = __builtin_amdgcn_readfirstlane(s_call);
  TR();

  const float scale = p.sx[0] * p.sw[0];
  const int xcd = b & 7;
  const int first = NPULL + ((xcd - NPULL) & 7);
  const unsigned cx = (unsigned)(GRID - first + 7) >> 3;
  unsigned* qh = (unsigned*)(ws + OFF_QUEUE) + xcd * 32;
  const int wn = wid >> 1, wm = wid & 1;
  const int fq = lane >> 4, fc = lane & 15;
  for (;;) {
    if (wid == 0) {
      unsigned traw = 0;
      if (lane == 0) traw = __hip_atomic_fetch_add(qh, 1u, __ATOMIC_RELAXED, __HIP_MEMORY_SCOPE_AGENT);
      const unsigned t = __builtin_amdgcn_readfirstlane(traw) - (call - 1) * (64u + cx);
      if (t < 64u) {
        const int j_ = t >> 4, d_ = (r + 1 + ((t >> 1) & 7)) & 7, nt_ = xcd * 2 + (t & 1);
        const unsigned* lf = land + (lane >> 3) * 256 + ((lane & 4) ? 128 + nt_ * 4 : (d_ * 4 + j_) * 4) + (lane & 3);
        const unsigned long long t0 = __builtin_amdgcn_s_memrealtime();
        for (;;) {
          const unsigned lv = ld_agent(lf), tv = ld_agent(tmo);
          if (tv == call) { if (lane == 0) s_ok = 0; break; }
          if (__ballot(lv < call) == 0ull) break;
          if (__builtin_amdgcn_s_memrealtime() - t0 > DEADLINE_TICKS) {
            if (lane == 0) { __hip_atomic_store(tmo, call, __ATOMIC_RELAXED, __HIP_MEMORY_SCOPE_AGENT); s_ok = 0; }
            break;
          }
          __builtin_amdgcn_s_sleep(2);
        }
        __builtin_amdgcn_fence(__ATOMIC_ACQUIRE, "agent");
        WAIT_VM0();
      }
      if (lane == 0) s_tile = t;
    }
    __syncthreads();
    const unsigned t = __builtin_amdgcn_readfirstlane(s_tile);
    if (t >= 64u) break;
    const int j = t >> 4, dd = (t >> 1) & 7, nt = xcd * 2 + (t & 1);
    const int d = (r + 1 + dd) & 7;
    const int mt = d * 4 + j;
    TR();
    f32x4 acc[4][4];
#pragma unroll
    for (int i = 0; i < 4; ++i)
#pragma unroll
      for (int k = 0; k < 4; ++k) acc[i][k] = (f32x4){0.f, 0.f, 0.f, 0.f};

    auto issue = [&](int s) {
      const int src = (r + (s >> 2)) & 7, kb = s & 3;
      const unsigned char* wb = (src == r) ? ws + OFF_WPUB + (size_t)(r + (call & 1) * 8) * MiB : ws + OFF_WLAND + (size_t)src * MiB;
      const unsigned char* xb = (src == r) ? ws + OFF_XPUB + (size_t)(call & 1) * 2 * MiB : ws + OFF_XLAND + (size_t)src * 2 * MiB;
      unsigned char* lb = lds + (s & 3) * 32768;
      const unsigned char* wsrc = wb + (size_t)(nt * 4 + kb) * 16384 + wid * 4096 + lane * 16;
      const unsigned char* xsrc = xb + (size_t)(mt * 4 + kb) * 16384 + wid * 4096 + lane * 16;
#pragma unroll
      for (int i = 0; i < 4; ++i)
        __builtin_amdgcn_global_load_lds((const unsigned*)(wsrc + i * 1024), (unsigned*)(lb + wid * 4096 + i * 1024), 16, 0, 0);
#pragma unroll
      for (int i = 0; i < 4; ++i)
        __builtin_amdgcn_global_load_lds((const unsigned*)(xsrc + i * 1024), (unsigned*)(lb + 16384 + wid * 4096 + i * 1024), 16, 0, 0);
    };
    auto rdfrag = [&](int s, i32x8* wf, i32x8* xf) {
      const unsigned char* lb = lds + (s & 3) * 32768 + lane * 16;
#pragma unroll
      for (int i = 0; i < 4; ++i) {
        wf[i] = ldfrag(lb + (wn * 4 + i) * 2048);
        xf[i] = ldfrag(lb + 16384 + (wm * 4 + i) * 2048);
      }
    };
    auto mma = [&](const i32x8* wf, const i32x8* xf) {
#pragma unroll
      for (int i = 0; i < 4; ++i)
#pragma unroll
        for (int k = 0; k < 4; ++k)
          acc[i][k] = __builtin_amdgcn_mfma_scale_f32_16x16x128_f8f6f4(wf[i], xf[k], acc[i][k], 0, 0, 0, 0x7F, 0, 0x7F);
    };
#define RAW_BAR() do { asm volatile("s_waitcnt lgkmcnt(0)" ::: "memory"); __builtin_amdgcn_s_barrier(); } while (0)
    i32x8 wfA[4], xfA[4], wfB[4], xfB[4];
    issue(0); issue(1); issue(2); issue(3);
    asm volatile("s_waitcnt vmcnt(24)" ::: "memory");
    RAW_BAR();
    rdfrag(0, wfA, xfA);
    for (int s = 0; s < 28; s += 2) {
      asm volatile("s_waitcnt vmcnt(16)" ::: "memory");
      RAW_BAR();
      issue(s + 4); rdfrag(s + 1, wfB, xfB); mma(wfA, xfA);
      asm volatile("s_waitcnt vmcnt(16)" ::: "memory");
      RAW_BAR();
      issue(s + 5); rdfrag(s + 2, wfA, xfA); mma(wfB, xfB);
    }
    asm volatile("s_waitcnt vmcnt(16)" ::: "memory");
    RAW_BAR();
    rdfrag(29, wfB, xfB); mma(wfA, xfA);
    asm volatile("s_waitcnt vmcnt(8)" ::: "memory");
    RAW_BAR();
    rdfrag(30, wfA, xfA); mma(wfB, xfB);
    asm volatile("s_waitcnt vmcnt(0)" ::: "memory");
    RAW_BAR();
    rdfrag(31, wfB, xfB); mma(wfA, xfA);
    mma(wfB, xfB);
    TR();

    const bool ok = s_ok != 0;
    if (d == r) {
      const __amdgpu_buffer_rsrc_t rs_out = mkrsrc(p.out);
      const unsigned ob = (unsigned)(((size_t)(j * 128) * NN + (size_t)r * NS + nt * 128) * 4);
#pragma unroll
      for (int i = 0; i < 4; ++i)
#pragma unroll
        for (int k = 0; k < 4; ++k) {
          f32x4 v = acc[i][k] * scale;
          if (!ok) v = (f32x4){__builtin_nanf(""), __builtin_nanf(""), __builtin_nanf(""), __builtin_nanf("")};
          const int m = (wm * 4 + k) * 16 + fc, n = (wn * 4 + i) * 16 + fq * 4;
          __builtin_amdgcn_raw_buffer_store_b128(__builtin_bit_cast(u32x4, v), rs_out, ob + (unsigned)(m * NN + n) * 4u, 0, OUTAUX);
        }
      RAW_BAR();
    } else {
      const __amdgpu_buffer_rsrc_t rs_loc = mkrsrc(ws);
      const unsigned ob = (unsigned)(OFF_OUT + ((size_t)(d * 4 + j) * 16 + nt) * 65536);
#pragma unroll
      for (int i = 0; i < 4; ++i)
#pragma unroll
        for (int k = 0; k < 4; ++k) {
          f32x4 v = acc[i][k] * scale;
          if (!ok) v = (f32x4){__builtin_nanf(""), __builtin_nanf(""), __builtin_nanf(""), __builtin_nanf("")};
          const int m = (wm * 4 + k) * 16 + fc, n = (wn * 4 + i) * 16 + fq * 4;
          __builtin_amdgcn_raw_buffer_store_b128(__builtin_bit_cast(u32x4, v), rs_loc, ob + (unsigned)(m * 128 + n) * 4u, 0, AUX_SYS);
        }
      WAIT_VM0();
      RAW_BAR();
      if (tid == 0) {
        unsigned* f = (unsigned*)(p.ws[d] + OFF_FLAGS) + r * FSTR + FLAG_O + j * 16 + nt;
        __hip_atomic_store(f, call, __ATOMIC_RELAXED, __HIP_MEMORY_SCOPE_SYSTEM);
      }
    }
    TR();
  }
}

}

extern "C" void kernel_launch(int rank, int nranks, ncclComm_t comm,
                              hipStream_t stream,
                              void* const* d_in, const int* in_sizes, int n_in,
                              void* d_out, int out_size,
                              const int* in_dtypes, int out_dtype,
                              void* const* d_out_all,
                              void* const* d_ws_all, size_t ws_bytes) {
  if (nranks != NR || ws_bytes < WS_NEED || rank < 0 || rank >= NR) {
    fprintf(stderr, "unexpected configuration\n");
    return;
  }
  Params p;
  p.x = (const unsigned char*)d_in[0];
  p.w = (const unsigned char*)d_in[1];
  p.sx = (const float*)d_in[2];
  p.sw = (const float*)d_in[3];
  p.out = (float*)d_out;
  for (int i = 0; i < NR; ++i) p.ws[i] = (unsigned char*)d_ws_all[i];
  p.rank = rank;
  hipLaunchKernelGGL(fused_kernel, dim3(GRID), dim3(NTHR), 0, stream, p);
  HIP_CHECK(hipGetLastError());
}
```

```cpp
#include <hip/hip_runtime.h>
#include <rccl/rccl.h>
#include <cstdio>
#include <cstdint>

#define HIP_CHECK(expr) do { hipError_t _e = (expr); if (_e != hipSuccess) { \
    fprintf(stderr, "HIP error %s at %s:%d\n", hipGetErrorString(_e), __FILE__, __LINE__); \
    return; } } while (0)

namespace {

typedef __attribute__((ext_vector_type(4))) unsigned u32x4;
typedef __attribute__((ext_vector_type(8))) int i32x8;
typedef __attribute__((ext_vector_type(4))) float f32x4;

constexpr int NR = 8;
constexpr int NN = 16384;
constexpr int KS = 512;
constexpr int NS = 2048;
#define OUTAUX 16
#define GPULL 6
constexpr int G = GPULL;
#define NWV 18
constexpr int NPULL = (NR - 1) * G;
constexpr int GRID = 256;
constexpr int NTHR = 256;

constexpr size_t MiB = 1ull << 20;
constexpr size_t OFF_CALLCNT = 0;
constexpr size_t OFF_FLAGS = 4096;
constexpr size_t OFF_LAND = 16384;
constexpr size_t OFF_QUEUE = 10240;
constexpr size_t OFF_TICKET = 11264;
constexpr size_t OFF_TMO = 12288;
constexpr size_t OFF_TRACE = 65536;
constexpr size_t OFF_XPUB = 1 * MiB;
constexpr size_t OFF_WPUB = 8 * MiB;
constexpr size_t OFF_XLAND = 32 * MiB;
constexpr size_t OFF_WLAND = 48 * MiB;
constexpr size_t OFF_OUT = 64 * MiB;
constexpr size_t WS_NEED = 96 * MiB;
constexpr unsigned OUT_BYTES = 512u * 16384u * 4u;

constexpr int FLAG_X0 = 0, FLAG_W = 32, FLAG_X = 48, FLAG_O = 72;
constexpr int FSTR = 192;
constexpr unsigned NBIG = 424;
constexpr unsigned NPIECE = NBIG + (448 - NBIG) * 4;
constexpr unsigned long long DEADLINE_TICKS = 50000000ull;

struct Params {
  const unsigned char* x;
  const unsigned char* w;
  const float* sx;
  const float* sw;
  float* out;
  unsigned char* ws[NR];
  int rank;
};

#define WAIT_VM0() asm volatile("s_waitcnt vmcnt(0)" ::: "memory")
#define TR() do {} while (0)
#define TR2(t) do {} while (0)

constexpr int AUX_SYS = 17;
constexpr int AUX_AGENT = 16;

__device__ __forceinline__ unsigned ld_sys(const unsigned* p) { return __hip_atomic_load(p, __ATOMIC_RELAXED, __HIP_MEMORY_SCOPE_SYSTEM); }
__device__ __forceinline__ unsigned ld_agent(const unsigned* p) { return __hip_atomic_load(p, __ATOMIC_RELAXED, __HIP_MEMORY_SCOPE_AGENT); }

__device__ __forceinline__ __amdgpu_buffer_rsrc_t mkrsrc(const void* base, unsigned bytes) {
  unsigned long long a = (unsigned long long)base;
  unsigned lo = __builtin_amdgcn_readfirstlane((unsigned)a), hi = __builtin_amdgcn_readfirstlane((unsigned)(a >> 32));
  void* pb = (void*)(((unsigned long long)hi << 32) | lo);
  return __builtin_amdgcn_make_buffer_rsrc(pb, 0, bytes, 0x00020000);
}

__device__ __forceinline__ bool wait_ge(const unsigned* p, unsigned target, unsigned* tmo, unsigned call) {
  unsigned long long t0 = __builtin_amdgcn_s_memrealtime();
  while (ld_sys(p) < target) {
    if (ld_agent(tmo) == call || __builtin_amdgcn_s_memrealtime() - t0 > DEADLINE_TICKS) {
      __hip_atomic_store(tmo, call, __ATOMIC_RELAXED, __HIP_MEMORY_SCOPE_AGENT);
      return false;
    }
    __builtin_amdgcn_s_sleep(2);
  }
  return true;
}

__device__ __forceinline__ void transpose4(unsigned r0, unsigned r1, unsigned r2, unsigned r3,
                                           unsigned& c0, unsigned& c1, unsigned& c2, unsigned& c3) {
  unsigned t0 = __builtin_amdgcn_perm(r1, r0, 0x05010400u);
  unsigned t1 = __builtin_amdgcn_perm(r1, r0, 0x07030602u);
  unsigned t2 = __builtin_amdgcn_perm(r3, r2, 0x05010400u);
  unsigned t3 = __builtin_amdgcn_perm(r3, r2, 0x07030602u);
  c0 = __builtin_amdgcn_perm(t2, t0, 0x05040100u);
  c1 = __builtin_amdgcn_perm(t2, t0, 0x07060302u);
  c2 = __builtin_amdgcn_perm(t3, t1, 0x05040100u);
  c3 = __builtin_amdgcn_perm(t3, t1, 0x07060302u);
}

__device__ __forceinline__ i32x8 ldfrag(const unsigned char* p) {
  const u32x4 a0 = *(const u32x4*)(p);
  const u32x4 a1 = *(const u32x4*)(p + 1024);
  return (i32x8){(int)a0[0], (int)a0[1], (int)a0[2], (int)a0[3], (int)a1[0], (int)a1[1], (int)a1[2], (int)a1[3]};
}

__global__ void __launch_bounds__(NTHR, 1) fused_kernel(Params p) {
  __shared__ __attribute__((aligned(16))) unsigned char lds[131072 + 64];
  volatile unsigned* sctl = (volatile unsigned*)(lds + 131072);
#define s_call sctl[0]
#define s_ok sctl[1]
#define s_tile sctl[2]
  const int tid = threadIdx.x, lane = tid & 63;
  const int wid = __builtin_amdgcn_readfirstlane(tid >> 6);
  const int b = blockIdx.x;
  const int r = p.rank;
  unsigned char* ws = p.ws[r];
  unsigned* tmo = (unsigned*)(ws + OFF_TMO);
  unsigned* land = (unsigned*)(ws + OFF_LAND);
  int trn = 0; (void)trn;
  TR();

  if (b < NPULL) {
    const int pi = b % (NR - 1);
    const unsigned src = (unsigned)(r + 1 + pi) % NR;
    if (src >= (unsigned)NR) return;
    const unsigned char* rws = p.ws[src];
    const unsigned* fl = (const unsigned*)(ws + OFF_FLAGS) + src * FSTR;
    unsigned* tk = (unsigned*)(ws + OFF_TICKET) + src * 32;
    int zopq = 0; asm volatile("" : "+v"(zopq));
    tk += zopq;
    const unsigned* f2p = lane == 63 ? tmo : fl + 128 + lane;
    unsigned cprev = 0, traw = 0;
    unsigned* cc = (unsigned*)(ws + OFF_CALLCNT) + b;
    if (tid == 0) cprev = ld_agent(cc);
    const bool active = wid * G + b / (NR - 1) < (NWV);
    if (lane == 0 && active) traw = __hip_atomic_fetch_add(tk, 1u, __ATOMIC_RELAXED, __HIP_MEMORY_SCOPE_AGENT);
    unsigned f0 = ld_sys(fl + lane), f1 = ld_sys(fl + 64 + lane), f2 = ld_sys(f2p);
    if (tid == 0) {
      __hip_atomic_store(cc, cprev + 1, __ATOMIC_RELAXED, __HIP_MEMORY_SCOPE_AGENT);
      s_call = cprev + 1;
    }
    __syncthreads();
    const unsigned call = __builtin_amdgcn_readfirstlane(s_call);
    if (!active) return;
    const __amdgpu_buffer_rsrc_t rs_rem = mkrsrc(rws, (unsigned)WS_NEED);
    const __amdgpu_buffer_rsrc_t rs_loc = mkrsrc(ws, (unsigned)WS_NEED);
    const __amdgpu_buffer_rsrc_t rs_out = mkrsrc(p.out, OUT_BYTES);
    bool ok = true;
    unsigned* pend = nullptr;
    const unsigned tbase = (call - 1) * (NPIECE + (unsigned)(NWV));
    const unsigned xpub = (unsigned)OFF_XPUB + (call & 1) * (unsigned)(2 * MiB), wpub = (unsigned)OFF_WPUB + (call & 1) * (unsigned)(8 * MiB);
    unsigned tn; unsigned long long rdy0, rdy1, rdy2;
#define NEXT_STATE() do { tn = __builtin_amdgcn_readfirstlane(traw) - tbase; rdy0 = __ballot(f0 >= call); rdy1 = __ballot(f1 >= call); \
      rdy2 = __ballot(lane != 63 && f2 >= call); if (__builtin_amdgcn_readlane(f2, 63) == call) ok = false; } while (0)
    NEXT_STATE();
    for (;;) {
      const unsigned t = tn;
      if (t >= NPIECE) break;
      if (lane == 0) traw = __hip_atomic_fetch_add(tk, 1u, __ATOMIC_RELAXED, __HIP_MEMORY_SCOPE_AGENT);
      unsigned fid, so, dof; unsigned* lc = nullptr; int b0 = 0, b1 = 4;
      if (t < 32u || (t >= 96u && t < 192u)) {
        const unsigned v = t < 32u ? t : t - 96u + 32u;
        const unsigned j = v >> 5, mt = ((v & 31) >> 2) * 4 + j;
        const unsigned o = mt * 65536u + (v & 3) * 16384u;
        fid = j == 0 ? FLAG_X0 + v : FLAG_X + (j - 1) * 8 + (mt >> 2); so = xpub + o; dof = (unsigned)(OFF_XLAND + (size_t)src * 2 * MiB) + o; lc = land + src * 256 + mt * 4 + (v & 3);
      } else if (t < 96u) {
        const unsigned u = t - 32u;
        fid = FLAG_W + (u >> 4) * 4 + (u & 3);
        so = wpub + (unsigned)r * (unsigned)MiB + u * 16384u; dof = (unsigned)(OFF_WLAND + (size_t)src * MiB) + u * 16384u; lc = land + src * 256 + 128 + u;
      } else {
        unsigned pc = t;
        if (t >= NBIG) { pc = NBIG + ((t - NBIG) >> 2); b0 = (t - NBIG) & 3; b1 = b0 + 1; }
        const unsigned v = pc - 192u, j = v >> 6, nt = (v >> 2) & 15, qt = v & 3;
        fid = FLAG_O + j * 16 + nt;
        so = (unsigned)(OFF_OUT + ((size_t)(r * 4 + j) * 16 + nt) * 65536) + qt * 16384u;
        dof = (unsigned)(((size_t)(j * 128 + qt * 32) * NN + (size_t)src * NS + nt * 128) * 4);
      }
      if (!(((fid < 64u ? rdy0 >> fid : fid < 128u ? rdy1 >> (fid - 64u) : rdy2 >> (fid - 128u)) & 1ull))) {
        if (pend) { WAIT_VM0(); if (lane == 0) __hip_atomic_store(pend, call, __ATOMIC_RELAXED, __HIP_MEMORY_SCOPE_AGENT); pend = nullptr; }
        ok = wait_ge(fl + fid, call, tmo, call) && ok;
      }
      f0 = ld_sys(fl + lane); f1 = ld_sys(fl + 64 + lane); f2 = ld_sys(f2p);
      if (t == 0u || t == 32u || t == 96u || t == 128u || t == 160u || t == 192u || t == 256u || t == 320u || t == 384u) TR2(t);
      if (lc) {
#pragma unroll
        for (int bt = 0; bt < 4; ++bt) {
          const unsigned o = (unsigned)bt * 4096u + lane * 16u;
          u32x4 v[4];
#pragma unroll
          for (int u = 0; u < 4; ++u) v[u] = __builtin_amdgcn_raw_buffer_load_b128(rs_rem, so + o + u * 1024u, 0, AUX_SYS);
          WAIT_VM0();
          if (bt == 3) NEXT_STATE();
          if (pend) { if (lane == 0) __hip_atomic_store(pend, call, __ATOMIC_RELAXED, __HIP_MEMORY_SCOPE_AGENT); pend = nullptr; }
#pragma unroll
          for (int u = 0; u < 4; ++u) __builtin_amdgcn_raw_buffer_store_b128(v[u], rs_loc, dof + o + u * 1024u, 0, AUX_AGENT);
        }
        pend = lc;
      } else {
        for (int bt = b0; bt < b1; ++bt) {
          u32x4 v[4];
          if (ok) {
#pragma unroll
            for (int u = 0; u < 4; ++u) v[u] = __builtin_amdgcn_raw_buffer_load_b128(rs_rem, so + bt * 4096u + u * 1024u + lane * 16u, 0, AUX_SYS);
          } else {
#pragma unroll
            for (int u = 0; u < 4; ++u) v[u] = (u32x4){0x7fc00000u, 0x7fc00000u, 0x7fc00000u, 0x7fc00000u};
          }
          WAIT_VM0();
          if (bt == b1 - 1) NEXT_STATE();
          if (pend) { if (lane == 0) __hip_atomic_store(pend, call, __ATOMIC_RELAXED, __HIP_MEMORY_SCOPE_AGENT); pend = nullptr; }
#pragma unroll
          for (int u = 0; u < 4; ++u) {
            const unsigned row = bt * 8 + u * 2 + (lane >> 5);
            __builtin_amdgcn_raw_buffer_store_b128(v[u], rs_out, dof + row * (unsigned)(NN * 4) + (lane & 31) * 16u, 0, OUTAUX);
          }
        }
      }
    }
    if (pend) { WAIT_VM0(); if (lane == 0) __hip_atomic_store(pend, call, __ATOMIC_RELAXED, __HIP_MEMORY_SCOPE_AGENT); }
    TR();
    return;
  }

  if (tid == 0) {
    unsigned* cc = (unsigned*)(ws + OFF_CALLCNT) + b;
    unsigned c = ld_agent(cc) + 1;
    __hip_atomic_store(cc, c, __ATOMIC_RELAXED, __HIP_MEMORY_SCOPE_AGENT);
    s_call = c; s_ok = 1;
  }
  const int gb = b - NPULL;
  if (gb < 184) {
    const __amdgpu_buffer_rsrc_t rs_loc = mkrsrc(ws, (unsigned)WS_NEED);
    if (gb >= 56) {
      const int u = gb - 56;
      const int kb = u & 3, nq = u >> 2, d = nq >> 2, ntq = nq & 3;
      const int ng = tid & 31, kg = tid >> 5;
      const unsigned char* sp = p.w + (size_t)(kb * 128 + kg * 16) * NN + nq * 512 + ng * 16;
      u32x4 in[16];
#pragma unroll
      for (int i = 0; i < 16; ++i) in[i] = *(const u32x4*)(sp + (size_t)i * NN);
      u32x4 o[16];
#pragma unroll
      for (int v = 0; v < 4; ++v)
#pragma unroll
        for (int w = 0; w < 4; ++w) {
          unsigned c0, c1, c2, c3;
          transpose4(in[4 * v][w], in[4 * v + 1][w], in[4 * v + 2][w], in[4 * v + 3][w], c0, c1, c2, c3);
          o[4 * w][v] = c0; o[4 * w + 1][v] = c1; o[4 * w + 2][v] = c2; o[4 * w + 3][v] = c3;
        }
      const int ntl = ng >> 3, nb = ng & 7, h = kg >> 2, q = kg & 3;
      const int base16 = ntl * 1024 + nb * 128 + h * 64 + q * 16;
#pragma unroll
      for (int c = 0; c < 16; ++c) *(u32x4*)(lds + (size_t)((base16 + c) ^ nb) * 16) = o[c];
      __syncthreads();
      const unsigned dof = (unsigned)(OFF_WPUB + (size_t)d * MiB) + (s_call & 1) * (unsigned)(8 * MiB);
#pragma unroll
      for (int i = 0; i < 16; ++i) {
        const int L = i * NTHR + tid;
        u32x4 v = *(const u32x4*)(lds + (size_t)(L ^ ((L >> 7) & 7)) * 16);
        const int nt = ntq * 4 + (L >> 10);
        __builtin_amdgcn_raw_buffer_store_b128(v, rs_loc, dof + (unsigned)(nt * 4 + kb) * 16384u + (unsigned)(L & 1023) * 16u, 0, AUX_SYS);
      }
      WAIT_VM0();
      __syncthreads();
      if (tid < 4) {
        const unsigned call = s_call;
        if (d == r) {
          __hip_atomic_store(land + r * 256 + 128 + (ntq * 4 + tid) * 4 + kb, call, __ATOMIC_RELAXED, __HIP_MEMORY_SCOPE_AGENT);
        } else if (tid == 0) {
          unsigned* f = (unsigned*)(p.ws[d] + OFF_FLAGS) + r * FSTR + FLAG_W + ntq * 4 + kb;
          __hip_atomic_store(f, call, __ATOMIC_RELAXED, __HIP_MEMORY_SCOPE_SYSTEM);
        }
      }
    } else if (gb >= 32) {
      const int j = 1 + ((gb - 32) >> 3), mt = ((gb - 32) & 7) * 4 + j;
      u32x4 v[16];
#pragma unroll
      for (int i = 0; i < 16; ++i) {
        const int L = i * NTHR + tid;
        const int kb = L >> 10, mb = (L >> 7) & 7, h = (L >> 6) & 1, q = (L >> 4) & 3, c = L & 15;
        v[i] = *(const u32x4*)(p.x + (size_t)(mt * 128 + mb * 16 + c) * KS + kb * 128 + h * 64 + q * 16);
      }
      __syncthreads();
      const unsigned xpub = (unsigned)OFF_XPUB + (s_call & 1) * (unsigned)(2 * MiB);
#pragma unroll
      for (int i = 0; i < 16; ++i)
        __builtin_amdgcn_raw_buffer_store_b128(v[i], rs_loc, xpub + (unsigned)mt * 65536u + (unsigned)(i * NTHR + tid) * 16u, 0, AUX_SYS);
      WAIT_VM0();
      __syncthreads();
      if (tid < NR + 4) {
        const unsigned call = s_call;
        if (tid >= NR) {
          __hip_atomic_store(land + r * 256 + mt * 4 + (tid - NR), call, __ATOMIC_RELAXED, __HIP_MEMORY_SCOPE_AGENT);
        } else if (tid != r) {
          unsigned* f = (unsigned*)(p.ws[tid] + OFF_FLAGS) + r * FSTR + FLAG_X + (gb - 32);
          __hip_atomic_store(f, call, __ATOMIC_RELAXED, __HIP_MEMORY_SCOPE_SYSTEM);
        }
      }
    } else {
      const int mt = (gb >> 2) * 4, kb = gb & 3;
      u32x4 v[4];
#pragma unroll
      for (int i = 0; i < 4; ++i) {
        const int L = i * NTHR + tid;
        const int mb = (L >> 7) & 7, h = (L >> 6) & 1, q = (L >> 4) & 3, c = L & 15;
        v[i] = *(const u32x4*)(p.x + (size_t)(mt * 128 + mb * 16 + c) * KS + kb * 128 + h * 64 + q * 16);
      }
      __syncthreads();
      const unsigned xpub = (unsigned)OFF_XPUB + (s_call & 1) * (unsigned)(2 * MiB);
#pragma unroll
      for (int i = 0; i < 4; ++i)
        __builtin_amdgcn_raw_buffer_store_b128(v[i], rs_loc, xpub + (unsigned)(mt * 4 + kb) * 16384u + (unsigned)(i * NTHR + tid) * 16u, 0, AUX_SYS);
      WAIT_VM0();
      __syncthreads();
      if (tid < NR) {
        const unsigned call = s_call;
        if (tid == r) {
          __hip_atomic_store(land + r * 256 + mt * 4 + kb, call, __ATOMIC_RELAXED, __HIP_MEMORY_SCOPE_AGENT);
        } else {
          unsigned* f = (unsigned*)(p.ws[tid] + OFF_FLAGS) + r * FSTR + FLAG_X0 + gb;
          __hip_atomic_store(f, call, __ATOMIC_RELAXED, __HIP_MEMORY_SCOPE_SYSTEM);
        }
      }
    }
  }
  __syncthreads();
  const unsigned call = __builtin_amdgcn_readfirstlane(s_call);
  TR();

  const float scale = p.sx[0] * p.sw[0];
  const int xcd = b & 7;
  const int first = NPULL + ((xcd - NPULL) & 7);
  const unsigned cx = (unsigned)(GRID - first + 7) >> 3;
  unsigned* qh = (unsigned*)(ws + OFF_QUEUE) + xcd * 32;
  const int wn = wid >> 1, wm = wid & 1;
  const int fq = lane >> 4, fc = lane & 15;
  for (;;) {
    if (wid == 0) {
      unsigned traw = 0;
      if (lane == 0) traw = __hip_atomic_fetch_add(qh, 1u, __ATOMIC_RELAXED, __HIP_MEMORY_SCOPE_AGENT);
      const unsigned t = __builtin_amdgcn_readfirstlane(traw) - (call - 1) * (64u + cx);
      if (t < 64u) {
        const int j_ = t >> 4, d_ = (r + 1 + ((t >> 1) & 7)) & 7, nt_ = xcd * 2 + (t & 1);
        const unsigned* lf = land + (lane >> 3) * 256 + ((lane & 4) ? 128 + nt_ * 4 : (d_ * 4 + j_) * 4) + (lane & 3);
        const unsigned long long t0 = __builtin_amdgcn_s_memrealtime();
        for (;;) {
          const unsigned lv = ld_agent(lf), tv = ld_agent(tmo);
          if (tv == call) { if (lane == 0) s_ok = 0; break; }
          if (__ballot(lv < call) == 0ull) break;
          if (__builtin_amdgcn_s_memrealtime() - t0 > DEADLINE_TICKS) {
            if (lane == 0) { __hip_atomic_store(tmo, call, __ATOMIC_RELAXED, __HIP_MEMORY_SCOPE_AGENT); s_ok = 0; }
            break;
          }
          __builtin_amdgcn_s_sleep(2);
        }
        __builtin_amdgcn_fence(__ATOMIC_ACQUIRE, "agent");
        WAIT_VM0();
      }
      if (lane == 0) s_tile = t;
    }
    __syncthreads();
    const unsigned t = __builtin_amdgcn_readfirstlane(s_tile);
    if (t >= 64u) break;
    const int j = t >> 4, dd = (t >> 1) & 7, nt = xcd * 2 + (t & 1);
    const int d = (r + 1 + dd) & 7;
    const int mt = d * 4 + j;
    TR();
    f32x4 acc[4][4];
#pragma unroll
    for (int i = 0; i < 4; ++i)
#pragma unroll
      for (int k = 0; k < 4; ++k) acc[i][k] = (f32x4){0.f, 0.f, 0.f, 0.f};

    auto issue = [&](int s) {
      const int src = (r + (s >> 2)) & 7, kb = s & 3;
      const unsigned char* wb = (src == r) ? ws + OFF_WPUB + (size_t)(r + (call & 1) * 8) * MiB : ws + OFF_WLAND + (size_t)src * MiB;
      const unsigned char* xb = (src == r) ? ws + OFF_XPUB + (size_t)(call & 1) * 2 * MiB : ws + OFF_XLAND + (size_t)src * 2 * MiB;
      unsigned char* lb = lds + (s & 3) * 32768;
      const unsigned char* wsrc = wb + (size_t)(nt * 4 + kb) * 16384 + wid * 4096 + lane * 16;
      const unsigned char* xsrc = xb + (size_t)(mt * 4 + kb) * 16384 + wid * 4096 + lane * 16;
#pragma unroll
      for (int i = 0; i < 4; ++i)
        __builtin_amdgcn_global_load_lds((const unsigned*)(wsrc + i * 1024), (unsigned*)(lb + wid * 4096 + i * 1024), 16, 0, 0);
#pragma unroll
      for (int i = 0; i < 4; ++i)
        __builtin_amdgcn_global_load_lds((const unsigned*)(xsrc + i * 1024), (unsigned*)(lb + 16384 + wid * 4096 + i * 1024), 16, 0, 0);
    };
    auto rdfrag = [&](int s, i32x8* wf, i32x8* xf) {
      const unsigned char* lb = lds + (s & 3) * 32768 + lane * 16;
#pragma unroll
      for (int i = 0; i < 4; ++i) {
        wf[i] = ldfrag(lb + (wn * 4 + i) * 2048);
        xf[i] = ldfrag(lb + 16384 + (wm * 4 + i) * 2048);
      }
    };
    auto mma = [&](const i32x8* wf, const i32x8* xf) {
#pragma unroll
      for (int i = 0; i < 4; ++i)
#pragma unroll
        for (int k = 0; k < 4; ++k)
          acc[i][k] = __builtin_amdgcn_mfma_scale_f32_16x16x128_f8f6f4(wf[i], xf[k], acc[i][k], 0, 0, 0, 0x7F, 0, 0x7F);
    };
#define RAW_BAR() do { asm volatile("s_waitcnt lgkmcnt(0)" ::: "memory"); __builtin_amdgcn_s_barrier(); } while (0)
    i32x8 wfA[4], xfA[4], wfB[4], xfB[4];
    issue(0); issue(1); issue(2); issue(3);
    asm volatile("s_waitcnt vmcnt(24)" ::: "memory");
    RAW_BAR();
    rdfrag(0, wfA, xfA);
    for (int s = 0; s < 28; s += 2) {
      asm volatile("s_waitcnt vmcnt(16)" ::: "memory");
      RAW_BAR();
      issue(s + 4); rdfrag(s + 1, wfB, xfB); mma(wfA, xfA);
      asm volatile("s_waitcnt vmcnt(16)" ::: "memory");
      RAW_BAR();
      issue(s + 5); rdfrag(s + 2, wfA, xfA); mma(wfB, xfB);
    }
    asm volatile("s_waitcnt vmcnt(16)" ::: "memory");
    RAW_BAR();
    rdfrag(29, wfB, xfB); mma(wfA, xfA);
    asm volatile("s_waitcnt vmcnt(8)" ::: "memory");
    RAW_BAR();
    rdfrag(30, wfA, xfA); mma(wfB, xfB);
    asm volatile("s_waitcnt vmcnt(0)" ::: "memory");
    RAW_BAR();
    rdfrag(31, wfB, xfB); mma(wfA, xfA);
    mma(wfB, xfB);
    TR();

    const bool ok = s_ok != 0;
    if (d == r) {
      const __amdgpu_buffer_rsrc_t rs_out = mkrsrc(p.out, OUT_BYTES);
      const unsigned ob = (unsigned)(((size_t)(j * 128) * NN + (size_t)r * NS + nt * 128) * 4);
#pragma unroll
      for (int i = 0; i < 4; ++i)
#pragma unroll
        for (int k = 0; k < 4; ++k) {
          f32x4 v = acc[i][k] * scale;
          if (!ok) v = (f32x4){__builtin_nanf(""), __builtin_nanf(""), __builtin_nanf(""), __builtin_nanf("")};
          const int m = (wm * 4 + k) * 16 + fc, n = (wn * 4 + i) * 16 + fq * 4;
          __builtin_amdgcn_raw_buffer_store_b128(__builtin_bit_cast(u32x4, v), rs_out, ob + (unsigned)(m * NN + n) * 4u, 0, OUTAUX);
        }
      RAW_BAR();
    } else {
      const __amdgpu_buffer_rsrc_t rs_loc = mkrsrc(ws, (unsigned)WS_NEED);
      const unsigned ob = (unsigned)(OFF_OUT + ((size_t)(d * 4 + j) * 16 + nt) * 65536);
#pragma unroll
      for (int i = 0; i < 4; ++i)
#pragma unroll
        for (int k = 0; k < 4; ++k) {
          f32x4 v = acc[i][k] * scale;
          if (!ok) v = (f32x4){__builtin_nanf(""), __builtin_nanf(""), __builtin_nanf(""), __builtin_nanf("")};
          const int m = (wm * 4 + k) * 16 + fc, n = (wn * 4 + i) * 16 + fq * 4;
          __builtin_amdgcn_raw_buffer_store_b128(__builtin_bit_cast(u32x4, v), rs_loc, ob + (unsigned)(m * 128 + n) * 4u, 0, AUX_SYS);
        }
      WAIT_VM0();
      RAW_BAR();
      if (tid == 0) {
        unsigned* f = (unsigned*)(p.ws[d] + OFF_FLAGS) + r * FSTR + FLAG_O + j * 16 + nt;
        __hip_atomic_store(f, call, __ATOMIC_RELAXED, __HIP_MEMORY_SCOPE_SYSTEM);
      }
    }
    TR();
  }
}

}

extern "C" void kernel_launch(int rank, int nranks, ncclComm_t comm,
                              hipStream_t stream,
                              void* const* d_in, const int* in_sizes, int n_in,
                              void* d_out, int out_size,
                              const int* in_dtypes, int out_dtype,
                              void* const* d_out_all,
                              void* const* d_ws_all, size_t ws_bytes) {
  if (nranks != NR || ws_bytes < WS_NEED || rank < 0 || rank >= NR || n_in < 4 || out_size != 512 * 16384) {
    fprintf(stderr, "unexpected configuration\n");
    return;
  }
  Params p;
  p.x = (const unsigned char*)d_in[0];
  p.w = (const unsigned char*)d_in[1];
  p.sx = (const float*)d_in[2];
  p.sw = (const float*)d_in[3];
  p.out = (float*)d_out;
  for (int i = 0; i < NR; ++i) p.ws[i] = (unsigned char*)d_ws_all[i];
  p.rank = rank;
  hipLaunchKernelGGL(fused_kernel, dim3(GRID), dim3(NTHR), 0, stream, p);
  HIP_CHECK(hipGetLastError());
}
```
